```python
import numpy as np

kernel_name = "gpu_a2a_gemm_m4096_k4096_n16384_e5m2e5m2_dqrelu"


ABSMAX_RTOL = 0.05

COMPUTE_DTYPE = "float32"
EPILOGUE = "dequant_relu"
IN_QUANT = None
OUT_QUANT = None
IN_FP8 = "e5m2e5m2"
IN_DTYPES = {"x": "e5m2", "w_mat": "e5m2",
             "scale_x": "float32", "scale_w": "float32"}
WORLD_SIZE = 8

def _snap_fp8(x, mant_bits, max_abs, min_normal):
    a = np.abs(x.astype(np.float64))
    _m, _e = np.frexp(a)
    q = np.where(a >= min_normal, np.ldexp(1.0, _e - 1 - mant_bits),
                 min_normal * 2.0 ** -mant_bits)
    snapped = np.minimum(np.round(a / q) * q, max_abs)
    return (np.sign(x) * snapped).astype(np.float32)

def setup_inputs():

    x = _snap_fp8(np.random.randn(4096, 4096).astype(np.float32), 2, 57344.0, 2.0 ** -14)
    w_mat = _snap_fp8(np.random.randn(4096, 16384).astype(np.float32), 2, 57344.0, 2.0 ** -14)
    scale_x = (0.02 * (1.0 + np.random.rand(1))).astype(np.float32)
    scale_w = (0.02 * (1.0 + np.random.rand(1))).astype(np.float32)
    return {"x": x, "w_mat": w_mat, "scale_x": scale_x, "scale_w": scale_w}

def reference(x, w_mat, scale_x, scale_w):
    acc = (x @ w_mat).astype(np.float32)
    y = (acc * (scale_x * scale_w)).astype(np.float32)
    return np.maximum(y, 0.0).astype(np.float32)
```

```cpp
#include <hip/hip_runtime.h>
#include <rccl/rccl.h>
#include <cstdio>
#define PUSHT0 2300
#define PUSHDT 2800
#define PACE 200
#define SPIN 600
#define SPIN0 200
#define QUIET 0
#define SLEEPN 1
#define PRIO 0
#define POLL1 1
#define NTHREADS 320

#define NPAYST 8
#define PUSHBITS "sc0 sc1"
#define LAS __attribute__((address_space(3)))
typedef int v2i __attribute__((ext_vector_type(2)));
typedef int v4i __attribute__((ext_vector_type(4)));
typedef int v8i __attribute__((ext_vector_type(8)));
typedef float v4f __attribute__((ext_vector_type(4)));

constexpr int NR = 8;
constexpr int ML = 512, NN = 16384, KK = 4096, KC = 512;
constexpr int BM = 256, BN = 128, BK = 128, NT = KK / BK;
constexpr int A_TILE = BM * BK, B_TILE = BK * BN, STAGE = A_TILE + B_TILE, NSTAGE = 3;
constexpr int LDS_CTRL = NSTAGE * STAGE;
constexpr int LDS_ORD = LDS_CTRL + 64;
constexpr int LDS_BYTES = LDS_ORD + 128;
constexpr size_t WS_CNT = 0;
constexpr size_t WS_TMO = 32768;
constexpr size_t WS_FLAG = 65536;
constexpr size_t WS_RECV = 1u << 20;
constexpr size_t RECV_BYTES = 2u * NR * KC * ML;
constexpr size_t WS_NEED = WS_RECV + RECV_BYTES;
constexpr unsigned long long DEADLINE_TICKS = 50000000ull;

struct Params {
  const unsigned char* x;
  const unsigned char* w;
  const float* sx;
  const float* sw;
  float* out;
  unsigned char* myws;
  int rank;
  int comm;
  unsigned char* ws[NR];
  unsigned long long* stamps;
};

#define SB __builtin_amdgcn_sched_barrier(0)
#define DMA(rs, ldsoff, voff, soff)                                                                      \
  asm volatile("s_mov_b32 m0, %0\n\ts_nop 0\n\tbuffer_load_dwordx4 %1, %2, %3 offen lds" ::"s"(lds0 + (ldsoff)), \
               "v"(voff), "s"(rs), "s"(soff) : "memory")

__device__ __forceinline__ v8i cat8(v4i a, v4i b) { return __builtin_shufflevector(a, b, 0, 1, 2, 3, 4, 5, 6, 7); }
__device__ __forceinline__ v8i cat8(v2i a, v2i b, v2i c, v2i d) {
  v4i lo = __builtin_shufflevector(a, b, 0, 1, 2, 3), hi = __builtin_shufflevector(c, d, 0, 1, 2, 3);
  return cat8(lo, hi);
}

__device__ __forceinline__ unsigned clk32() { unsigned long long t; asm volatile("s_memtime %0\n\ts_waitcnt lgkmcnt(0)" : "=s"(t) :: "memory"); return (unsigned)t; }
__device__ __forceinline__ unsigned long long rtc64() { unsigned long long t; asm volatile("s_memrealtime %0\n\ts_waitcnt lgkmcnt(0)" : "=s"(t) :: "memory"); return t; }

__global__ __launch_bounds__(320, 2) void fused_a2a_gemm(Params P) {
  extern __shared__ __attribute__((aligned(16))) char smem_g[];
  LAS char* smem = (LAS char*)smem_g;
  const int tid = threadIdx.x, lane = tid & 63;
  const int wave = __builtin_amdgcn_readfirstlane(tid >> 6);
  const int b = blockIdx.x;
  const int rank = P.rank;
  unsigned char* myws = P.myws;

  if (wave == 4) {
    __builtin_amdgcn_s_setprio(3);
    const unsigned cstart = clk32();
#define NPUSHWG 224
    const bool pusher = P.comm && b < NPUSHWG && (P.comm != 3 || rank < 2) && (P.comm != 4 || ((rank + 1 + (b % 56) % 7) & 7) < 2);
    const int rnd = b / 56, bb = b % 56, d = 1 + bb % 7, slice = bb / 7;
    const unsigned peer = (unsigned)(rank + d) & 7u;
    v4i pay[8];
    if (pusher) {
      const unsigned char* src = P.x + ((size_t)(peer * ML + slice * 64 + (lane >> 3)) * KC + rnd * 128 + (lane & 7) * 16);
#pragma unroll
      for (int i = 0; i < 8; ++i) pay[i] = *(const v4i*)(src + i * 8 * KC);
    }
    unsigned* cnt = (unsigned*)(myws + WS_CNT) + b * 32;
    unsigned* tmo = (unsigned*)(myws + WS_TMO);
    const unsigned call = __builtin_amdgcn_readfirstlane(__hip_atomic_load(cnt, __ATOMIC_RELAXED, __HIP_MEMORY_SCOPE_AGENT)) + 1u;
    unsigned char* pws = P.ws[peer];
    unsigned char* dst = pws + WS_RECV + ((size_t)(((call & 1u) * NR + rank) * 4 + rnd) * (ML * 128) + slice * 64 * 128 + lane * 16);
    unsigned* pflag = (unsigned*)(pws + WS_FLAG) + (lane & 7) * 256 + (rank * 8 + slice) * 4 + rnd;
    const unsigned char* myflag = myws + WS_FLAG + (b & 7) * 1024 + lane * 16;
    unsigned push_c = PUSHT0 + PUSHDT * rnd;
    const int us = lane / 7, up = (rank + 7 - lane % 7) & 7;
    const int uq = 4 * up + us;
#define CSTAMP(i)
    constexpr unsigned ALL = (1u << 28) - 1u;
    bool bad = false;
    unsigned arrived = P.comm == 1 ? 0u : ALL, used = 0u;
    int nass = 0;
    int ps = pusher ? 0 : 2;
    v4i f = {0, 0, 0, 0};
    bool polled = false;
    int issued = 0, seq_st = 0, seq_poll = 0, nst = 0;
#define PUSH1(i) asm volatile("global_store_dwordx4 %0, %1, off " PUSHBITS "\n\ts_nop 1" ::"v"(dst + (i) * 1024), "v"(pay[i]) : "memory")
#define PUSH_PAYLOAD                                                                                                    \
    _Pragma("unroll") for (int i = 0; i < NPAYST; ++i)                                                                  \
      asm volatile("global_store_dwordx4 %0, %1, off " PUSHBITS "\n\ts_nop 1" ::"v"(dst + i * 1024), "v"(pay[i]) : "memory")
    *(LAS unsigned*)(smem + LDS_CTRL + 4 * (lane & 1)) = (lane & 1) ? 0u : call;
    CSTAMP(0);
    asm volatile("s_waitcnt vmcnt(0)" : "+v"(pay[0]), "+v"(pay[1]), "+v"(pay[2]), "+v"(pay[3]), "+v"(pay[4]), "+v"(pay[5]), "+v"(pay[6]), "+v"(pay[7]) :: "memory");
    if (pusher && rnd == 0) { PUSH_PAYLOAD; ps = 1; issued = seq_st = 8; CSTAMP(1); }
    __hip_atomic_store(cnt, call, __ATOMIC_RELAXED, __HIP_MEMORY_SCOPE_AGENT);
    ++issued;
    for (int j = 0;; ++j) {
      const int need = j < 3 ? 0 : (j < 29 ? j - 1 : 28);
      const unsigned c0 = clk32();
      unsigned long long t0 = 0;
      bool pslot = false;
      for (;;) {
        unsigned ib; asm volatile("s_getreg_b32 %0, hwreg(HW_REG_IB_STS)" : "=s"(ib));
        const int done = issued - (int)((ib & 0xfu) | ((ib >> 18) & 0x30u));
        if (ps == 1 && done >= seq_st) {
          if (issued - seq_st >= 2) asm volatile("s_waitcnt vmcnt(2)" ::: "memory");
          else if (issued - seq_st == 1) asm volatile("s_waitcnt vmcnt(1)" ::: "memory");
          else asm volatile("s_waitcnt vmcnt(0)" ::: "memory");
          __hip_atomic_store(pflag, call, __ATOMIC_RELAXED, __HIP_MEMORY_SCOPE_SYSTEM);
          ++issued; ps = 2; if (rnd == 0) CSTAMP(2); else CSTAMP(3);
        }
        if (polled && done >= seq_poll) {
          polled = false;
          asm volatile("" : "+v"(f) :: "memory");
          const unsigned long long m0_ = __ballot((int)(f.x - call) >= 0), m1_ = __ballot((int)(f.y - call) >= 0),
                                   m2_ = __ballot((int)(f.z - call) >= 0), m3_ = __ballot((int)(f.w - call) >= 0);
          const unsigned long long mm = us == 0 ? m0_ : us == 1 ? m1_ : us == 2 ? m2_ : m3_;
          arrived = (unsigned)__ballot(lane < 28 && ((mm >> (8 * up)) & 0xffull) == 0xffull);
        }
        if (ps == 0 && clk32() - cstart >= push_c) {
          switch (nst) {
            case 0: PUSH1(0); break; case 1: PUSH1(1); break; case 2: PUSH1(2); break; case 3: PUSH1(3); break;
            case 4: PUSH1(4); break; case 5: PUSH1(5); break; case 6: PUSH1(6); break; default: PUSH1(7); break;
          }
          ++issued; push_c += PACE;
          if (++nst == 8) { ps = 1; seq_st = issued; }
        }
        if (!polled && arrived != ALL && j >= 1 && (nass < need || !POLL1 || !pslot)) {
          pslot = true;
          asm volatile("global_load_dwordx4 %0, %1, off sc0 sc1" : "=v"(f) : "v"(myflag) : "memory");
          ++issued; seq_poll = issued; polled = true;
        }
        while (nass < need && (arrived & ~used) != 0u) {
          const int u = __builtin_ctz(arrived & ~used);
          used |= 1u << u;
          *(LAS int*)(smem + LDS_ORD + 16 + 4 * nass) = __builtin_amdgcn_readlane(uq, u);
          ++nass;
        }
        const bool idle = (ps == 2 || (ps == 0 && c0 - cstart + SPIN < push_c)) && (arrived == ALL || j < 1 || (POLL1 && pslot));
        if (nass >= need && idle) break;
        const unsigned dc = clk32() - c0;
        if (nass >= need) { if (dc > (j == 0 && rnd == 0 ? SPIN0 : SPIN)) break; }
        else if (dc > 50000u || t0 != 0) {
          const unsigned long long tn = rtc64();
          if (t0 == 0) t0 = tn;
          if (__builtin_amdgcn_readfirstlane(__hip_atomic_load(tmo, __ATOMIC_RELAXED, __HIP_MEMORY_SCOPE_AGENT)) == call || tn - t0 > DEADLINE_TICKS) {
            __hip_atomic_store(tmo, call, __ATOMIC_RELAXED, __HIP_MEMORY_SCOPE_AGENT);
            ++issued; bad = true; arrived = ALL; polled = false;
          }
          ++issued;
          __builtin_amdgcn_s_sleep(8);
        }
        __builtin_amdgcn_s_sleep(SLEEPN);
      }
      const bool last = arrived == ALL && ps == 2 && !polled;
      if (last) {
        const unsigned rest = ALL & ~used;
        const bool mineleft = lane < 28 && ((rest >> (lane & 31)) & 1u);
        *(LAS int*)(smem + LDS_ORD + (mineleft ? 16 + 4 * (nass + __builtin_popcount(rest & ((1u << (lane & 31)) - 1u))) : 0)) = uq;
        if (P.comm && __builtin_amdgcn_readfirstlane(__hip_atomic_load(tmo, __ATOMIC_RELAXED, __HIP_MEMORY_SCOPE_AGENT)) == call) bad = true;
        if (bad) *(LAS unsigned*)(smem + LDS_CTRL + 4) = 1u;
      }
      asm volatile("s_waitcnt lgkmcnt(0)\n\ts_barrier" ::: "memory");
      if (j == 0) __builtin_amdgcn_s_setprio(PRIO);
      if (last) {
        break;
      }
    }
    return;
  }

  const int xcd = b & 7, idx = b >> 3;
  const int ntile = (idx >> 1) * 8 + ((xcd + (idx >> 1)) & 7);
  const int n0 = ntile * BN, m0 = (idx & 1) * BM;
  const int wm = wave >> 1, wn = wave & 1;
  const unsigned lds0 = (unsigned)(size_t)smem;

  const __amdgpu_buffer_rsrc_t rsW = __builtin_amdgcn_make_buffer_rsrc((void*)P.w, 0, KK * NN, 0x00020000);
  const __amdgpu_buffer_rsrc_t rsX = __builtin_amdgcn_make_buffer_rsrc((void*)P.x, 0, KK * KC, 0x00020000);
  const __amdgpu_buffer_rsrc_t rsR = __builtin_amdgcn_make_buffer_rsrc((void*)(myws + WS_RECV), 0, (int)RECV_BYTES, 0x00020000);
  const int swzA = ((lane & 7) ^ (((lane >> 4) + 4 * (wave & 1)) & 7)) * 16;
  const int voffAx = (lane >> 3) * KC + swzA;
  const int voffAr = (lane >> 3) * 128 + swzA;
  const int t16 = lane & 15, g4 = lane >> 4;
  const int voffB = (lane >> 3) * NN + (((lane & 7) ^ (((lane >> 4) & 3) | (((wave >> 1) & 1) << 2))) * 16);

#define ISSUE_B(i, st, kb) DMA(rsW, (st) + A_TILE + (4 * (i) + wave) * 1024, voffB, (kb) + (4 * (i) + wave) * 8 * NN)
#define ISSUE_AX(i, st, ab) DMA(rsX, (st) + (4 * (i) + wave) * 1024, voffAx, (ab) + (4 * (i) + wave) * 8 * KC)
#define ISSUE_AR(i, st, ab) DMA(rsR, (st) + (4 * (i) + wave) * 1024, voffAr, (ab) + (4 * (i) + wave) * 8 * 128)
#define STAMP(i)

  const int ax0 = (rank * ML + m0) * KC;
  const int wb0 = rank * KC * NN + n0;
#pragma unroll
  for (int T = 0; T < 3; ++T) {
#pragma unroll
    for (int i = 0; i < 4; ++i) ISSUE_B(i, T * STAGE, wb0 + T * 128 * NN);
    if (T < 2) {
#pragma unroll
      for (int i = 0; i < 8; ++i) ISSUE_AX(i, T * STAGE, ax0 + T * 128);
    }
  }
  int adA[2], adB[4];
#pragma unroll
  for (int e = 0; e < 2; ++e) adA[e] = (wm * 128 + t16) * 128 + 16 * ((4 * e + g4) ^ ((t16 >> 1) & 7));
#pragma unroll
  for (int nb = 0; nb < 4; ++nb)
    adB[nb] = A_TILE + (16 * g4 + (t16 >> 1)) * 128 + 16 * ((4 * wn + nb) ^ (((t16 >> 2) & 3) | ((g4 & 1) << 2))) + 8 * (t16 & 1);
  v4f acc[8][4];
#pragma unroll
  for (int i = 0; i < 8; ++i)
#pragma unroll
    for (int j = 0; j < 4; ++j) { acc[i][j] = (v4f){0.f, 0.f, 0.f, 0.f}; asm volatile("" : "+a"(acc[i][j])); }
  SB;
  asm volatile("s_waitcnt vmcnt(16)\n\ts_barrier" ::: "memory");
  const unsigned call = __builtin_amdgcn_readfirstlane(*(LAS unsigned*)(smem + LDS_CTRL));
  const int rbase = (int)(call & 1u) * (NR * KC * ML) + m0 * 128;

  int st = 0;
  const int qdef = (4 * rank + 28) & 31;
  int ab = ax0 + 2 * 128;
  const LAS int* ord = (const LAS int*)(smem + LDS_ORD) + 3;
#define IA(i) do { if (AK == 0) { ISSUE_AX(i, stp, ab); } else if (AK == 1) { ISSUE_AR(i, stp, ab); } } while (0)
#define IL(i) do { if (LATE) { ISSUE_AR(i, stp, abl); } } while (0)
#define IB(i) do { if (BKD < 3) { ISSUE_B(i, st, kb); } } while (0)

  v4i aw[4][2];
  v2i bc[4][4], bn[4][4];
#define RDA(slot, mb, st) do { aw[slot][0] = *(const LAS v4i*)(smem + (st) + adA[0] + (mb) * 2048);            \
                               aw[slot][1] = *(const LAS v4i*)(smem + (st) + adA[1] + (mb) * 2048); } while (0)
#define RDB(dst, nb, i, st) \
  dst[nb][i] = __builtin_amdgcn_ds_read_tr8_b64_v2i32((LAS v2i*)(smem + (st) + adB[nb] + ((i) >> 1) * 8192 + ((i) & 1) * 1024))
#define RDB2(dst, nb, i, st) do { if (W >= 0) { RDB(dst, nb, i, st); RDB(dst, nb, (i) + 1, st); } } while (0)
#define RDAN(slot, mb) do { if (W >= 0) { RDA(slot, mb, stn); } } while (0)
#define MM(B_, mb, nb)                                                                                            \
  SB; asm volatile("v_mfma_f32_16x16x128_f8f6f4 %0, %1, %2, %0 cbsz:1 blgp:1" : "+a"(acc[mb][nb])                  \
                   : "v"(cat8(B_[nb][0], B_[nb][1], B_[nb][2], B_[nb][3])), "v"(cat8(aw[(mb) & 3][0], aw[(mb) & 3][1]))); SB
#pragma unroll
  for (int nb = 0; nb < 4; ++nb) { RDB(bc, nb, 0, 0); RDB(bc, nb, 1, 0); RDB(bc, nb, 2, 0); RDB(bc, nb, 3, 0); if (nb == 0) RDA(0, 0, 0); }
  RDA(1, 1, 0); RDA(2, 2, 0); RDA(3, 3, 0);
#define ITERX(AK_, BKD_, W_, LATE_, BC, BNX)                                                                     \
  do {                                                                                                          \
    constexpr int AK = AK_, BKD = BKD_, W = W_; constexpr bool LATE = LATE_;                                    \
    const int stn = st == 2 * STAGE ? 0 : st + STAGE;                                                           \
    const int stp = st == 0 ? 2 * STAGE : st - STAGE;                                                           \
    MM(BC, 0, 0); IA(0); MM(BC, 0, 1); MM(BC, 0, 2); MM(BC, 0, 3); RDA(0, 4, st); IA(1);                         \
    MM(BC, 1, 0); MM(BC, 1, 1); IA(2); MM(BC, 1, 2); MM(BC, 1, 3); RDA(1, 5, st);                                \
    MM(BC, 2, 0); IA(3); MM(BC, 2, 1); MM(BC, 2, 2); IA(4); MM(BC, 2, 3); RDA(2, 6, st);                         \
    MM(BC, 3, 0); MM(BC, 3, 1); IA(5); MM(BC, 3, 2); MM(BC, 3, 3); RDA(3, 7, st); IA(6);                         \
    MM(BC, 4, 0); MM(BC, 4, 1); IA(7); MM(BC, 4, 2); MM(BC, 4, 3);                                               \
    if (W == 12) asm volatile("s_waitcnt vmcnt(12) lgkmcnt(0)\n\ts_barrier" ::: "memory");                      \
    else if (W == 0) asm volatile("s_waitcnt vmcnt(0) lgkmcnt(0)\n\ts_barrier" ::: "memory");                   \
    int qv = 0, qlv = 0;                                                                                        \
    if (LATE) qlv = ord[-1];                                                                                    \
    if (BKD == 2) qv = ord[0];                                                                                  \
    ++ord;                                                                                                      \
    RDAN(0, 0);                                                                                                 \
    MM(BC, 5, 0); RDB2(BNX, 0, 0, stn); MM(BC, 5, 1); RDB2(BNX, 0, 2, stn);                                      \
    const int q = BKD == 0 ? 4 * rank + 3 : BKD == 1 ? qdef : __builtin_amdgcn_readfirstlane(qv);               \
    const int ql = LATE ? __builtin_amdgcn_readfirstlane(qlv) : 0;                                              \
    const int kb = (q << 21) + n0, abn = BKD == 0 ? ax0 + 3 * 128 : rbase + (q << 16), abl = rbase + (ql << 16); \
    if (LATE && ql != qdef) {                                                                                   \
      const int kbl = (ql << 21) + n0;                                                                          \
      ISSUE_B(0, stp, kbl); ISSUE_B(1, stp, kbl); ISSUE_B(2, stp, kbl); ISSUE_B(3, stp, kbl);                    \
    }                                                                                                           \
    IL(0); IL(1);                                                                                               \
    MM(BC, 5, 2); IL(2); RDB2(BNX, 1, 0, stn); if (!LATE) IB(0); MM(BC, 5, 3); IL(3); RDAN(1, 1);                \
    MM(BC, 6, 0); IL(4); RDB2(BNX, 1, 2, stn); MM(BC, 6, 1); IL(5); RDB2(BNX, 2, 0, stn); if (!LATE) IB(1);      \
    MM(BC, 6, 2); IL(6); RDB2(BNX, 2, 2, stn); MM(BC, 6, 3); IL(7); RDAN(2, 2); if (LATE) IB(0);                 \
    MM(BC, 7, 0); RDB2(BNX, 3, 0, stn); if (LATE) IB(1); else IB(2); MM(BC, 7, 1); RDB2(BNX, 3, 2, stn); if (LATE) IB(2); \
    MM(BC, 7, 2); IB(3); MM(BC, 7, 3); RDAN(3, 3); SB;                                                          \
    st = stn; ab = abn;                                                                                         \
  } while (0)
#define ITER(AK, BKD, W, LATE) ITERX(AK, BKD, W, LATE, bc, bn)
#define ITER2(AK, BKD, W, LATE) ITERX(AK, BKD, W, LATE, bn, bc)

  ITER(0, 0, 12, false);
  ITER2(0, 1, 12, false);
  ITER(2, 2, 0, true);
  ITER2(1, 2, 12, false);
#pragma unroll 1
  for (int t = 4; t < 28; t += 2) { ITER(1, 2, 12, false); ITER2(1, 2, 12, false); }
  ITER(1, 2, 12, false);
  ITER2(1, 3, 12, false);
  ITER(2, 3, 0, false);
  ITER2(2, 3, -1, false);

#define PIN4(i) "+a"(acc[i][0]), "+a"(acc[i][1]), "+a"(acc[i][2]), "+a"(acc[i][3])
  asm volatile("s_nop 15\n\ts_nop 7" : PIN4(7), PIN4(6), PIN4(5), PIN4(4));
  asm volatile("" : PIN4(3), PIN4(2), PIN4(1), PIN4(0));
  SB;
  STAMP(2);
  const float scale = P.sx[0] * P.sw[0];
  const bool bad = *(LAS unsigned*)(smem + LDS_CTRL + 4) != 0u;
#define FIN(y_) (bad ? __builtin_nanf("") : ((y_) < 0.f ? 0.f : (y_)))
  asm volatile("s_waitcnt lgkmcnt(0)\n\ts_barrier" ::: "memory");
  LAS char* img = smem + wave * 32768;
  float* obase = P.out + (size_t)(m0 + wm * 128 + (lane >> 4)) * NN + n0 + wn * 64 + (lane & 15) * 4;
#pragma unroll
  for (int mq = 0; mq < 4; ++mq) {
#pragma unroll
    for (int mh = 0; mh < 2; ++mh)
#pragma unroll
      for (int nb = 0; nb < 4; ++nb) {
        v4f v;
#pragma unroll
        for (int e = 0; e < 4; ++e) { const float y = acc[2 * mq + mh][nb][e] * scale; v[e] = FIN(y); }
        *(LAS v4f*)(img + (mq * 32 + mh * 16 + t16) * 256 + (((nb * 4 + g4) ^ (t16 & 7)) * 16)) = v;
      }
#pragma unroll
    for (int i = 0; i < 8; ++i) {
      const int row = mq * 32 + i * 4 + (lane >> 4);
      const v4f v = *(const LAS v4f*)(img + row * 256 + (((lane & 15) ^ (row & 7)) * 16));
      float* ptr = obase + (size_t)(mq * 32 + i * 4) * NN;
      asm volatile("global_store_dwordx4 %0, %1, off sc0 sc1\n\ts_nop 1" ::"v"(ptr), "v"(v) : "memory");
    }
  }
}

#define HIP_CHECK(expr) do { hipError_t _e = (expr); if (_e != hipSuccess) { \
    fprintf(stderr, "HIP error %s at %s:%d\n", hipGetErrorString(_e), __FILE__, __LINE__); \
    return; } } while (0)

extern "C" void kernel_launch(int rank, int nranks, ncclComm_t comm, hipStream_t stream,
                              void* const* d_in, const int* in_sizes, int n_in,
                              void* d_out, int out_size, const int* in_dtypes, int out_dtype,
                              void* const* d_out_all, void* const* d_ws_all, size_t ws_bytes) {
  if (nranks != NR || (unsigned)rank >= (unsigned)NR || n_in != 4 || ws_bytes < WS_NEED || out_size != ML * NN ||
      in_sizes[0] != KK * KC || in_sizes[1] != KK * NN) {
    fprintf(stderr, "kernel_launch: unexpected problem shape\n");
    return;
  }
  static bool configured = false;
  if (!configured) {
    HIP_CHECK(hipFuncSetAttribute((const void*)fused_a2a_gemm, hipFuncAttributeMaxDynamicSharedMemorySize, LDS_BYTES));
    configured = true;
  }
  Params P{};
  P.x = (const unsigned char*)d_in[0];
  P.w = (const unsigned char*)d_in[1];
  P.sx = (const float*)d_in[2];
  P.sw = (const float*)d_in[3];
  P.out = (float*)d_out;
  for (int p = 0; p < NR; ++p) P.ws[p] = (unsigned char*)d_ws_all[p];
  P.rank = rank;
  P.myws = P.ws[rank];
  P.comm = 1;
  fused_a2a_gemm<<<256, NTHREADS, LDS_BYTES, stream>>>(P);
  HIP_CHECK(hipGetLastError());
}
```

```cpp
#include <hip/hip_runtime.h>
#include <rccl/rccl.h>
#include <cstdio>
#define PUSHT0 2300
#define PUSHDT 2800
#define PACE 0
#define SPIN 600
#define SPIN0 200
#define QUIET 0
#define SLEEPN 1
#define PRIO 0
#define POLL1 1
#define NTHREADS 384
#define PFMIN 5
#define PFMODE 1
#define COLDC 4500
#define PFOUT 2
#define PFSPIN 0

#define NPAYST 8
#define PFBITS ""
#define WBITS ""
#define OUTBITS "sc0 sc1"
#define PUSHBITS "sc0 sc1"
#define LAS __attribute__((address_space(3)))
typedef int v2i __attribute__((ext_vector_type(2)));
typedef int v4i __attribute__((ext_vector_type(4)));
typedef int v8i __attribute__((ext_vector_type(8)));
typedef float v4f __attribute__((ext_vector_type(4)));

constexpr int NR = 8;
constexpr int ML = 512, NN = 16384, KK = 4096, KC = 512;
constexpr int BM = 256, BN = 128, BK = 128, NT = KK / BK;
constexpr int A_TILE = BM * BK, B_TILE = BK * BN, STAGE = A_TILE + B_TILE, NSTAGE = 3;
constexpr int NBAR = NT + 1;
constexpr int LDS_CTRL = NSTAGE * STAGE;
constexpr int LDS_ORD = LDS_CTRL + 64;
constexpr int LDS_SCRAP = LDS_ORD + 128;
constexpr int LDS_BYTES = LDS_SCRAP + 256;
constexpr size_t WS_CNT = 0;
constexpr size_t WS_TMO = 32768;
constexpr size_t WS_FLAG = 65536;
constexpr size_t WS_RECV = 1u << 20;
constexpr size_t RECV_BYTES = 2u * NR * KC * ML;
constexpr size_t WS_NEED = WS_RECV + RECV_BYTES;
constexpr unsigned long long DEADLINE_TICKS = 50000000ull;

struct Params {
  const unsigned char* x;
  const unsigned char* w;
  const float* sx;
  const float* sw;
  float* out;
  unsigned char* myws;
  int rank;
  int comm;
  unsigned char* ws[NR];
  unsigned long long* stamps;
};

#define SB __builtin_amdgcn_sched_barrier(0)
#define DMA(rs, ldsoff, voff, soff)                                                                      \
  asm volatile("s_mov_b32 m0, %0\n\ts_nop 0\n\tbuffer_load_dwordx4 %1, %2, %3 offen lds" ::"s"(lds0 + (ldsoff)), \
               "v"(voff), "s"(rs), "s"(soff) : "memory")

__device__ __forceinline__ v8i cat8(v4i a, v4i b) { return __builtin_shufflevector(a, b, 0, 1, 2, 3, 4, 5, 6, 7); }
__device__ __forceinline__ v8i cat8(v2i a, v2i b, v2i c, v2i d) {
  v4i lo = __builtin_shufflevector(a, b, 0, 1, 2, 3), hi = __builtin_shufflevector(c, d, 0, 1, 2, 3);
  return cat8(lo, hi);
}

__device__ __forceinline__ unsigned clk32() { unsigned long long t; asm volatile("s_memtime %0\n\ts_waitcnt lgkmcnt(0)" : "=s"(t) :: "memory"); return (unsigned)t; }
__device__ __forceinline__ unsigned long long rtc64() { unsigned long long t; asm volatile("s_memrealtime %0\n\ts_waitcnt lgkmcnt(0)" : "=s"(t) :: "memory"); return t; }

__global__ __launch_bounds__(384, 2) void fused_a2a_gemm(Params P) {
  extern __shared__ __attribute__((aligned(16))) char smem_g[];
  LAS char* smem = (LAS char*)smem_g;
  const int tid = threadIdx.x, lane = tid & 63;
  const int wave = __builtin_amdgcn_readfirstlane(tid >> 6);
  const int b = blockIdx.x;
  const int rank = P.rank;
  unsigned char* myws = P.myws;

  const int xcd = b & 7, idx = b >> 3;
  const int ntile = (idx >> 1) * 8 + ((xcd + (idx >> 1)) & 7);
  const int n0 = ntile * BN, m0 = (idx & 1) * BM;

  if (wave == 5) {
    const __amdgpu_buffer_rsrc_t rsW = __builtin_amdgcn_make_buffer_rsrc((void*)P.w, 0, KK * NN, 0x00020000);
    const __amdgpu_buffer_rsrc_t rsX = __builtin_amdgcn_make_buffer_rsrc((void*)P.x, 0, KK * KC, 0x00020000);
    const __amdgpu_buffer_rsrc_t rsR = __builtin_amdgcn_make_buffer_rsrc((void*)(myws + WS_RECV), 0, (int)RECV_BYTES, 0x00020000);
    const unsigned scrap = (unsigned)(size_t)smem + LDS_SCRAP;
#define PF(rs, voff, soff, bits) asm volatile("s_mov_b32 m0, %0\n\ts_nop 0\n\tbuffer_load_dword %1, %2, %3 offen " bits " lds" ::"s"(scrap), "v"(voff), "s"(rs), "s"(soff) : "memory")
    const int vw = b * 8192 + lane * 128;
    const int arow = m0 + (idx >> 1) * 16 + (lane & 15);
    const int vx = (rank * ML + arow) * KC + 128 * (1 + (lane >> 4) % 3), vr = arow * 128;
    int qw = 4 * rank + 2;
    int nw = 3;
#define NEXTQ(q_, n_) ((n_) < 4 ? (q_) + 1 : (n_) == 4 ? ((4 * rank + 28) & 31) : ((((q_) + 28) & 31) >> 2) == rank ? (((q_) + 24) & 31) + 1 : ((q_) + 28) & 31)
#define PFW() do { qw = NEXTQ(qw, nw); ++nw; PF(rsW, vw, qw << 21, PFBITS); } while (0)
    const unsigned cstart = clk32();
    bool cold = false;
    for (int j = 0; j < NBAR; ++j) {
      asm volatile("s_barrier" ::: "memory");
      const unsigned c0 = clk32();
      if (j == 0) cold = c0 - cstart > COLDC;
      while (cold) {
        while (nw < j + PFMIN && nw < NT) { qw = NEXTQ(qw, nw); ++nw; }
        if (nw >= NT) { cold = false; break; }
        unsigned ib; asm volatile("s_getreg_b32 %0, hwreg(HW_REG_IB_STS)" : "=s"(ib));
        if ((ib & 0xC0000Fu) < PFOUT) PFW();
        const unsigned blocked = __builtin_amdgcn_readfirstlane(*(volatile LAS unsigned*)(smem + LDS_CTRL + 12));
        if (!blocked && clk32() - c0 > PFSPIN) break;
        __builtin_amdgcn_s_sleep(1);
      }
    }
    asm volatile("s_waitcnt vmcnt(0)" ::: "memory");
    return;
  }

  if (wave == 4) {
    __builtin_amdgcn_s_setprio(3);
    const unsigned cstart = clk32();
#define NPUSHWG 224
    const bool pusher = P.comm && b < NPUSHWG && (P.comm != 3 || rank < 2) && (P.comm != 4 || ((rank + 1 + (b % 56) % 7) & 7) < 2);
    const int rnd = b / 56, bb = b % 56, d = 1 + bb % 7, slice = bb / 7;
    const unsigned peer = (unsigned)(rank + d) & 7u;
    v4i pay[8];
    if (pusher) {
      const unsigned char* src = P.x + ((size_t)(peer * ML + slice * 64 + (lane >> 3)) * KC + rnd * 128 + (lane & 7) * 16);
#pragma unroll
      for (int i = 0; i < 8; ++i) pay[i] = *(const v4i*)(src + i * 8 * KC);
    }
    unsigned* cnt = (unsigned*)(myws + WS_CNT) + b * 32;
    unsigned* tmo = (unsigned*)(myws + WS_TMO);
    const unsigned call = __builtin_amdgcn_readfirstlane(__hip_atomic_load(cnt, __ATOMIC_RELAXED, __HIP_MEMORY_SCOPE_AGENT)) + 1u;
    unsigned char* pws = P.ws[peer];
    unsigned char* dst = pws + WS_RECV + ((size_t)(((call & 1u) * NR + rank) * 4 + rnd) * (ML * 128) + slice * 64 * 128 + lane * 16);
    unsigned* pflag = (unsigned*)(pws + WS_FLAG) + (lane & 7) * 256 + (rank * 8 + slice) * 4 + rnd;
    const unsigned char* myflag = myws + WS_FLAG + (b & 7) * 1024 + lane * 16;
    unsigned push_c = PUSHT0 + PUSHDT * rnd;
    const int us = lane / 7, up = (rank + 7 - lane % 7) & 7;
    const int uq = 4 * up + us;
#define CSTAMP(i)
    constexpr unsigned ALL = (1u << 28) - 1u;
    bool bad = false;
    unsigned arrived = P.comm == 1 ? 0u : ALL, used = 0u;
    int nass = 0;
    int ps = pusher ? 0 : 2;
    v4i f = {0, 0, 0, 0};
    bool polled = false, blk = false;
    int issued = 0, seq_st = 0, seq_poll = 0, nst = 0;
#define PUSH1(i) asm volatile("global_store_dwordx4 %0, %1, off " PUSHBITS "\n\ts_nop 1" ::"v"(dst + (i) * 1024), "v"(pay[i]) : "memory")
#define PUSH_PAYLOAD                                                                                                    \
    _Pragma("unroll") for (int i = 0; i < NPAYST; ++i)                                                                  \
      asm volatile("global_store_dwordx4 %0, %1, off " PUSHBITS "\n\ts_nop 1" ::"v"(dst + i * 1024), "v"(pay[i]) : "memory")
    *(LAS unsigned*)(smem + LDS_CTRL + 4 * (lane & 3)) = (lane & 3) == 0 ? call : (lane & 3) == 2 ? arrived : 0u;
    CSTAMP(0);
    asm volatile("s_waitcnt vmcnt(0)" : "+v"(pay[0]), "+v"(pay[1]), "+v"(pay[2]), "+v"(pay[3]), "+v"(pay[4]), "+v"(pay[5]), "+v"(pay[6]), "+v"(pay[7]) :: "memory");
    if (pusher && rnd == 0) { PUSH_PAYLOAD; ps = 1; issued = seq_st = 8; CSTAMP(1); }
    __hip_atomic_store(cnt, call, __ATOMIC_RELAXED, __HIP_MEMORY_SCOPE_AGENT);
    ++issued;
    for (int j = 0;; ++j) {
      const int need = j < 3 ? 0 : (j < 29 ? j - 1 : 28);
      const unsigned c0 = clk32();
      unsigned long long t0 = 0;
      bool pslot = false;
      for (;;) {
        unsigned ib; asm volatile("s_getreg_b32 %0, hwreg(HW_REG_IB_STS)" : "=s"(ib));
        const int done = issued - (int)((ib & 0xfu) | ((ib >> 18) & 0x30u));
        if (ps == 1 && done >= seq_st) {
          if (issued - seq_st >= 2) asm volatile("s_waitcnt vmcnt(2)" ::: "memory");
          else if (issued - seq_st == 1) asm volatile("s_waitcnt vmcnt(1)" ::: "memory");
          else asm volatile("s_waitcnt vmcnt(0)" ::: "memory");
          __hip_atomic_store(pflag, call, __ATOMIC_RELAXED, __HIP_MEMORY_SCOPE_SYSTEM);
          ++issued; ps = 2; if (rnd == 0) CSTAMP(2); else CSTAMP(3);
        }
        if (polled && done >= seq_poll) {
          polled = false;
          asm volatile("" : "+v"(f) :: "memory");
          const unsigned long long m0_ = __ballot((int)(f.x - call) >= 0), m1_ = __ballot((int)(f.y - call) >= 0),
                                   m2_ = __ballot((int)(f.z - call) >= 0), m3_ = __ballot((int)(f.w - call) >= 0);
          const unsigned long long mm = us == 0 ? m0_ : us == 1 ? m1_ : us == 2 ? m2_ : m3_;
          arrived = (unsigned)__ballot(lane < 28 && ((mm >> (8 * up)) & 0xffull) == 0xffull);
          *(LAS unsigned*)(smem + LDS_CTRL + 8) = arrived;
        }
        if (ps == 0 && clk32() - cstart >= push_c) {
          PUSH_PAYLOAD; issued += 8; ps = 1; seq_st = issued;
        }
        if (!polled && arrived != ALL && j >= 1 && (nass < need || !POLL1 || !pslot)) {
          pslot = true;
          asm volatile("global_load_dwordx4 %0, %1, off sc0 sc1" : "+v"(f) : "v"(myflag) : "memory");
          ++issued; seq_poll = issued; polled = true;
        }
        while (nass < need && (arrived & ~used) != 0u) {
          const int u = __builtin_ctz(arrived & ~used);
          used |= 1u << u;
          *(LAS int*)(smem + LDS_ORD + 16 + 4 * nass) = __builtin_amdgcn_readlane(uq, u);
          ++nass;
        }
        const bool idle = (ps == 2 || (ps == 0 && c0 - cstart + SPIN < push_c)) && (arrived == ALL || j < 1 || (POLL1 && pslot));
        if (nass >= need && idle) break;
        if (nass < need && !blk) { blk = true; *(LAS unsigned*)(smem + LDS_CTRL + 12) = 1u; }
        if (nass >= need && blk) { blk = false; *(LAS unsigned*)(smem + LDS_CTRL + 12) = 0u; }
        const unsigned dc = clk32() - c0;
        if (nass >= need) { if (dc > (j == 0 && rnd == 0 ? SPIN0 : SPIN)) break; }
        else if (dc > 50000u || t0 != 0) {
          const unsigned long long tn = rtc64();
          if (t0 == 0) t0 = tn;
          if (__builtin_amdgcn_readfirstlane(__hip_atomic_load(tmo, __ATOMIC_RELAXED, __HIP_MEMORY_SCOPE_AGENT)) == call || tn - t0 > DEADLINE_TICKS) {
            __hip_atomic_store(tmo, call, __ATOMIC_RELAXED, __HIP_MEMORY_SCOPE_AGENT);
            ++issued; bad = true; arrived = ALL; polled = false;
          }
          ++issued;
          __builtin_amdgcn_s_sleep(8);
        }
        __builtin_amdgcn_s_sleep(SLEEPN);
      }
      if (blk) { blk = false; *(LAS unsigned*)(smem + LDS_CTRL + 12) = 0u; }
      const bool last = (arrived == ALL && ps == 2 && !polled) || j == NBAR - 2;
      if (last) {
        const unsigned rest = ALL & ~used;
        const bool mineleft = lane < 28 && ((rest >> (lane & 31)) & 1u);
        *(LAS int*)(smem + LDS_ORD + (mineleft ? 16 + 4 * (nass + __builtin_popcount(rest & ((1u << (lane & 31)) - 1u))) : 0)) = uq;
        if (P.comm && __builtin_amdgcn_readfirstlane(__hip_atomic_load(tmo, __ATOMIC_RELAXED, __HIP_MEMORY_SCOPE_AGENT)) == call) bad = true;
        if (bad) *(LAS unsigned*)(smem + LDS_CTRL + 4) = 1u;
      }
      asm volatile("s_waitcnt lgkmcnt(0)\n\ts_barrier" ::: "memory");
      if (j == 0) __builtin_amdgcn_s_setprio(PRIO);
      if (last) {
        for (++j; j < NBAR; ++j) {
          asm volatile("s_barrier" ::: "memory");
        }
        break;
      }
    }
    asm volatile("s_waitcnt vmcnt(0)" : "+v"(f) :: "memory");
    if (ps == 1) {
      __hip_atomic_store(pflag, call, __ATOMIC_RELAXED, __HIP_MEMORY_SCOPE_SYSTEM);
    }
    return;
  }

  const int wm = wave >> 1, wn = wave & 1;
  const unsigned lds0 = (unsigned)(size_t)smem;

  const __amdgpu_buffer_rsrc_t rsW = __builtin_amdgcn_make_buffer_rsrc((void*)P.w, 0, KK * NN, 0x00020000);
  const __amdgpu_buffer_rsrc_t rsX = __builtin_amdgcn_make_buffer_rsrc((void*)P.x, 0, KK * KC, 0x00020000);
  const __amdgpu_buffer_rsrc_t rsR = __builtin_amdgcn_make_buffer_rsrc((void*)(myws + WS_RECV), 0, (int)RECV_BYTES, 0x00020000);
  const int swzA = ((lane & 7) ^ (((lane >> 4) + 4 * (wave & 1)) & 7)) * 16;
  const int voffAx = (lane >> 3) * KC + swzA;
  const int voffAr = (lane >> 3) * 128 + swzA;
  const int t16 = lane & 15, g4 = lane >> 4;
  const int voffB = (lane >> 3) * NN + (((lane & 7) ^ (((lane >> 4) & 3) | (((wave >> 1) & 1) << 2))) * 16);

#define DMAW(rs, ldsoff, voff, soff)                                                                     \
  asm volatile("s_mov_b32 m0, %0\n\ts_nop 0\n\tbuffer_load_dwordx4 %1, %2, %3 offen " WBITS " lds" ::"s"(lds0 + (ldsoff)), \
               "v"(voff), "s"(rs), "s"(soff) : "memory")
#define ISSUE_B(i, st, kb) DMAW(rsW, (st) + A_TILE + (4 * (i) + wave) * 1024, voffB, (kb) + (4 * (i) + wave) * 8 * NN)
#define ISSUE_AX(i, st, ab) DMA(rsX, (st) + (4 * (i) + wave) * 1024, voffAx, (ab) + (4 * (i) + wave) * 8 * KC)
#define ISSUE_AR(i, st, ab) DMA(rsR, (st) + (4 * (i) + wave) * 1024, voffAr, (ab) + (4 * (i) + wave) * 8 * 128)
#define STAMP(i)

  const int ax0 = (rank * ML + m0) * KC;
  const int wb0 = rank * KC * NN + n0;
#pragma unroll
  for (int T = 0; T < 3; ++T) {
#pragma unroll
    for (int i = 0; i < 4; ++i) ISSUE_B(i, T * STAGE, wb0 + T * 128 * NN);
    if (T < 2) {
#pragma unroll
      for (int i = 0; i < 8; ++i) ISSUE_AX(i, T * STAGE, ax0 + T * 128);
    }
  }
  int adA[2], adB[4];
#pragma unroll
  for (int e = 0; e < 2; ++e) adA[e] = (wm * 128 + t16) * 128 + 16 * ((4 * e + g4) ^ ((t16 >> 1) & 7));
#pragma unroll
  for (int nb = 0; nb < 4; ++nb)
    adB[nb] = A_TILE + (16 * g4 + (t16 >> 1)) * 128 + 16 * ((4 * wn + nb) ^ (((t16 >> 2) & 3) | ((g4 & 1) << 2))) + 8 * (t16 & 1);
  v4f acc[8][4];
#pragma unroll
  for (int i = 0; i < 8; ++i)
#pragma unroll
    for (int j = 0; j < 4; ++j) { acc[i][j] = (v4f){0.f, 0.f, 0.f, 0.f}; asm volatile("" : "+a"(acc[i][j])); }
  SB;
  asm volatile("s_waitcnt vmcnt(16)\n\ts_barrier" ::: "memory");
  const unsigned call = __builtin_amdgcn_readfirstlane(*(LAS unsigned*)(smem + LDS_CTRL));
  const int rbase = (int)(call & 1u) * (NR * KC * ML) + m0 * 128;

  int st = 0;
  const int qdef = (4 * rank + 28) & 31;
  int ab = ax0 + 2 * 128;
  const LAS int* ord = (const LAS int*)(smem + LDS_ORD) + 3;
#define IA(i) do { if (AK == 0) { ISSUE_AX(i, stp, ab); } else if (AK == 1) { ISSUE_AR(i, stp, ab); } } while (0)
#define IL(i) do { if (LATE) { ISSUE_AR(i, stp, abl); } } while (0)
#define IB(i) do { if (BKD < 3) { ISSUE_B(i, st, kb); } } while (0)

  v4i aw[4][2];
  v2i bc[4][4], bn[4][4];
#define RDA(slot, mb, st) do { aw[slot][0] = *(const LAS v4i*)(smem + (st) + adA[0] + (mb) * 2048);            \
                               aw[slot][1] = *(const LAS v4i*)(smem + (st) + adA[1] + (mb) * 2048); } while (0)
#define RDB(dst, nb, i, st) \
  dst[nb][i] = __builtin_amdgcn_ds_read_tr8_b64_v2i32((LAS v2i*)(smem + (st) + adB[nb] + ((i) >> 1) * 8192 + ((i) & 1) * 1024))
#define RDB2(dst, nb, i, st) do { if (W >= 0) { RDB(dst, nb, i, st); RDB(dst, nb, (i) + 1, st); } } while (0)
#define RDAN(slot, mb) do { if (W >= 0) { RDA(slot, mb, stn); } } while (0)
#define MM(B_, mb, nb)                                                                                            \
  SB; asm volatile("v_mfma_f32_16x16x128_f8f6f4 %0, %1, %2, %0 cbsz:1 blgp:1" : "+a"(acc[mb][nb])                  \
                   : "v"(cat8(B_[nb][0], B_[nb][1], B_[nb][2], B_[nb][3])), "v"(cat8(aw[(mb) & 3][0], aw[(mb) & 3][1]))); SB
#pragma unroll
  for (int nb = 0; nb < 4; ++nb) { RDB(bc, nb, 0, 0); RDB(bc, nb, 1, 0); RDB(bc, nb, 2, 0); RDB(bc, nb, 3, 0); if (nb == 0) RDA(0, 0, 0); }
  RDA(1, 1, 0); RDA(2, 2, 0); RDA(3, 3, 0);
#define ITERX(AK_, BKD_, W_, LATE_, BC, BNX)                                                                     \
  do {                                                                                                          \
    constexpr int AK = AK_, BKD = BKD_, W = W_; constexpr bool LATE = LATE_;                                    \
    const int stn = st == 2 * STAGE ? 0 : st + STAGE;                                                           \
    const int stp = st == 0 ? 2 * STAGE : st - STAGE;                                                           \
    MM(BC, 0, 0); IA(0); MM(BC, 0, 1); MM(BC, 0, 2); MM(BC, 0, 3); RDA(0, 4, st); IA(1);                         \
    MM(BC, 1, 0); MM(BC, 1, 1); IA(2); MM(BC, 1, 2); MM(BC, 1, 3); RDA(1, 5, st);                                \
    MM(BC, 2, 0); IA(3); MM(BC, 2, 1); MM(BC, 2, 2); IA(4); MM(BC, 2, 3); RDA(2, 6, st);                         \
    MM(BC, 3, 0); MM(BC, 3, 1); IA(5); MM(BC, 3, 2); MM(BC, 3, 3); RDA(3, 7, st); IA(6);                         \
    MM(BC, 4, 0); MM(BC, 4, 1); IA(7); MM(BC, 4, 2); MM(BC, 4, 3);                                               \
    if (W == 12) asm volatile("s_waitcnt vmcnt(12) lgkmcnt(0)\n\ts_barrier" ::: "memory");                      \
    else if (W == 0) asm volatile("s_waitcnt vmcnt(0) lgkmcnt(0)\n\ts_barrier" ::: "memory");                   \
    int qv = 0, qlv = 0;                                                                                        \
    if (LATE) qlv = ord[-1];                                                                                    \
    if (BKD == 2) qv = ord[0];                                                                                  \
    ++ord;                                                                                                      \
    RDAN(0, 0);                                                                                                 \
    MM(BC, 5, 0); RDB2(BNX, 0, 0, stn); MM(BC, 5, 1); RDB2(BNX, 0, 2, stn);                                      \
    const int q = BKD == 0 ? 4 * rank + 3 : BKD == 1 ? qdef : __builtin_amdgcn_readfirstlane(qv);               \
    const int ql = LATE ? __builtin_amdgcn_readfirstlane(qlv) : 0;                                              \
    const int kb = (q << 21) + n0, abn = BKD == 0 ? ax0 + 3 * 128 : rbase + (q << 16), abl = rbase + (ql << 16); \
    if (LATE && ql != qdef) {                                                                                   \
      const int kbl = (ql << 21) + n0;                                                                          \
      ISSUE_B(0, stp, kbl); ISSUE_B(1, stp, kbl); ISSUE_B(2, stp, kbl); ISSUE_B(3, stp, kbl);                    \
    }                                                                                                           \
    IL(0); IL(1);                                                                                               \
    MM(BC, 5, 2); IL(2); RDB2(BNX, 1, 0, stn); if (!LATE) IB(0); MM(BC, 5, 3); IL(3); RDAN(1, 1);                \
    MM(BC, 6, 0); IL(4); RDB2(BNX, 1, 2, stn); MM(BC, 6, 1); IL(5); RDB2(BNX, 2, 0, stn); if (!LATE) IB(1);      \
    MM(BC, 6, 2); IL(6); RDB2(BNX, 2, 2, stn); MM(BC, 6, 3); IL(7); RDAN(2, 2); if (LATE) IB(0);                 \
    MM(BC, 7, 0); RDB2(BNX, 3, 0, stn); if (LATE) IB(1); else IB(2); MM(BC, 7, 1); RDB2(BNX, 3, 2, stn); if (LATE) IB(2); \
    MM(BC, 7, 2); IB(3); MM(BC, 7, 3); RDAN(3, 3); SB;                                                          \
    st = stn; ab = abn;                                                                                         \
  } while (0)
#define ITER(AK, BKD, W, LATE) ITERX(AK, BKD, W, LATE, bc, bn)
#define ITER2(AK, BKD, W, LATE) ITERX(AK, BKD, W, LATE, bn, bc)

  ITER(0, 0, 12, false);
  ITER2(0, 1, 12, false);
  ITER(2, 2, 0, true);
  ITER2(1, 2, 12, false);
#pragma unroll 1
  for (int t = 4; t < 28; t += 2) { ITER(1, 2, 12, false); ITER2(1, 2, 12, false); }
  ITER(1, 2, 12, false);
  ITER2(1, 3, 12, false);
  ITER(2, 3, 0, false);
  ITER2(2, 3, -1, false);

#define PIN4(i) "+a"(acc[i][0]), "+a"(acc[i][1]), "+a"(acc[i][2]), "+a"(acc[i][3])
  asm volatile("s_nop 15\n\ts_nop 7" : PIN4(7), PIN4(6), PIN4(5), PIN4(4));
  asm volatile("" : PIN4(3), PIN4(2), PIN4(1), PIN4(0));
  SB;
  STAMP(2);
  const float scale = P.sx[0] * P.sw[0];
  const bool bad = *(LAS unsigned*)(smem + LDS_CTRL + 4) != 0u;
#define FIN(y_) (bad ? __builtin_nanf("") : ((y_) < 0.f ? 0.f : (y_)))
  asm volatile("s_waitcnt lgkmcnt(0)\n\ts_barrier" ::: "memory");
  LAS char* img = smem + wave * 32768;
  float* obase = P.out + (size_t)(m0 + wm * 128 + (lane >> 4)) * NN + n0 + wn * 64 + (lane & 15) * 4;
#pragma unroll
  for (int mq = 0; mq < 4; ++mq) {
#pragma unroll
    for (int mh = 0; mh < 2; ++mh)
#pragma unroll
      for (int nb = 0; nb < 4; ++nb) {
        v4f v;
#pragma unroll
        for (int e = 0; e < 4; ++e) { const float y = acc[2 * mq + mh][nb][e] * scale; v[e] = FIN(y); }
        *(LAS v4f*)(img + (mq * 32 + mh * 16 + t16) * 256 + (((nb * 4 + g4) ^ (t16 & 7)) * 16)) = v;
      }
#pragma unroll
    for (int i = 0; i < 8; ++i) {
      const int row = mq * 32 + i * 4 + (lane >> 4);
      const v4f v = *(const LAS v4f*)(img + row * 256 + (((lane & 15) ^ (row & 7)) * 16));
      float* ptr = obase + (size_t)(mq * 32 + i * 4) * NN;
      asm volatile("global_store_dwordx4 %0, %1, off " OUTBITS "\n\ts_nop 1" ::"v"(ptr), "v"(v) : "memory");
    }
  }
}

#define HIP_CHECK(expr) do { hipError_t _e = (expr); if (_e != hipSuccess) { \
    fprintf(stderr, "HIP error %s at %s:%d\n", hipGetErrorString(_e), __FILE__, __LINE__); \
    return; } } while (0)

extern "C" void kernel_launch(int rank, int nranks, ncclComm_t comm, hipStream_t stream,
                              void* const* d_in, const int* in_sizes, int n_in,
                              void* d_out, int out_size, const int* in_dtypes, int out_dtype,
                              void* const* d_out_all, void* const* d_ws_all, size_t ws_bytes) {
  if (nranks != NR || (unsigned)rank >= (unsigned)NR || n_in != 4 || ws_bytes < WS_NEED || out_size != ML * NN ||
      in_sizes[0] != KK * KC || in_sizes[1] != KK * NN) {
    fprintf(stderr, "kernel_launch: unexpected problem shape\n");
    return;
  }
  static bool configured = false;
  if (!configured) {
    HIP_CHECK(hipFuncSetAttribute((const void*)fused_a2a_gemm, hipFuncAttributeMaxDynamicSharedMemorySize, LDS_BYTES));
    configured = true;
  }
  Params P{};
  P.x = (const unsigned char*)d_in[0];
  P.w = (const unsigned char*)d_in[1];
  P.sx = (const float*)d_in[2];
  P.sw = (const float*)d_in[3];
  P.out = (float*)d_out;
  for (int p = 0; p < NR; ++p) P.ws[p] = (unsigned char*)d_ws_all[p];
  P.rank = rank;
  P.myws = P.ws[rank];
  P.comm = 1;
  fused_a2a_gemm<<<256, NTHREADS, LDS_BYTES, stream>>>(P);
  HIP_CHECK(hipGetLastError());
}
```

```cpp
#include <hip/hip_runtime.h>
#include <rccl/rccl.h>
#include <cstdio>
#define PUSHT0 2300
#define PUSHDT 2800
#define PACE 0
#define SPIN 600
#define SPIN0 200
#define QUIET 0
#define SLEEPN 1
#define PRIO 0
#define POLL1 1
#define NTHREADS 384
#define PFMODE 0
#define PFSH 1
#define PFNB 0
#define TLBW 0
#define COLDC 4500
#define PFD 16
#define PFD0 1

#define NPAYST 8
#define PFBITS ""
#define WBITS ""
#define OUTBITS "sc0 sc1"
#define PUSHBITS "sc0 sc1"
#define LAS __attribute__((address_space(3)))
typedef int v2i __attribute__((ext_vector_type(2)));
typedef int v4i __attribute__((ext_vector_type(4)));
typedef int v8i __attribute__((ext_vector_type(8)));
typedef float v4f __attribute__((ext_vector_type(4)));

constexpr int NR = 8;
constexpr int ML = 512, NN = 16384, KK = 4096, KC = 512;
constexpr int BM = 256, BN = 128, BK = 128, NT = KK / BK;
constexpr int A_TILE = BM * BK, B_TILE = BK * BN, STAGE = A_TILE + B_TILE, NSTAGE = 3;
constexpr int NBAR = NT + 1;
constexpr int LDS_CTRL = NSTAGE * STAGE;
constexpr int LDS_ORD = LDS_CTRL + 64;
constexpr int LDS_SCRAP = LDS_ORD + 128;
constexpr int LDS_BYTES = LDS_SCRAP + 1024;
constexpr size_t WS_CNT = 0;
constexpr size_t WS_TMO = 32768;
constexpr size_t WS_FLAG = 65536;
constexpr size_t WS_RECV = 1u << 20;
constexpr int PAR = NR * KC * ML + (512 << 10);
constexpr size_t RECV_BYTES = (size_t)PAR + NR * KC * ML;
constexpr size_t WS_NEED = 6u << 20;
constexpr unsigned long long DEADLINE_TICKS = 50000000ull;

struct Params {
  const unsigned char* x;
  const unsigned char* w;
  const float* sx;
  const float* sw;
  float* out;
  unsigned char* myws;
  int rank;
  int comm;
  unsigned char* ws[NR];
  unsigned long long* stamps;
};

#define SB __builtin_amdgcn_sched_barrier(0)
#define DMA(rs, ldsoff, voff, soff)                                                                      \
  asm volatile("s_mov_b32 m0, %0\n\ts_nop 0\n\tbuffer_load_dwordx4 %1, %2, %3 offen lds" ::"s"(lds0 + (ldsoff)), \
               "v"(voff), "s"(rs), "s"(soff) : "memory")

__device__ __forceinline__ v8i cat8(v4i a, v4i b) { return __builtin_shufflevector(a, b, 0, 1, 2, 3, 4, 5, 6, 7); }
__device__ __forceinline__ v8i cat8(v2i a, v2i b, v2i c, v2i d) {
  v4i lo = __builtin_shufflevector(a, b, 0, 1, 2, 3), hi = __builtin_shufflevector(c, d, 0, 1, 2, 3);
  return cat8(lo, hi);
}

__device__ __forceinline__ unsigned clk32() { unsigned long long t; asm volatile("s_memtime %0\n\ts_waitcnt lgkmcnt(0)" : "=s"(t) :: "memory"); return (unsigned)t; }
__device__ __forceinline__ unsigned long long rtc64() { unsigned long long t; asm volatile("s_memrealtime %0\n\ts_waitcnt lgkmcnt(0)" : "=s"(t) :: "memory"); return t; }

__global__ __launch_bounds__(384, 2) void fused_a2a_gemm(Params P) {
  extern __shared__ __attribute__((aligned(16))) char smem_g[];
  LAS char* smem = (LAS char*)smem_g;
  const int tid = threadIdx.x, lane = tid & 63;
  const int wave = __builtin_amdgcn_readfirstlane(tid >> 6);
  const int b = blockIdx.x;
  const int rank = P.rank;
  unsigned char* myws = P.myws;

  const int xcd = b & 7, idx = b >> 3;
  const int ntile = (idx >> 1) * 8 + ((xcd + (idx >> 1)) & 7);
  const int n0 = ntile * BN, m0 = (idx & 1) * BM;

  if (wave == 5) {
    const __amdgpu_buffer_rsrc_t rsW = __builtin_amdgcn_make_buffer_rsrc((void*)P.w, 0, KK * NN, 0x00020000);
    const __amdgpu_buffer_rsrc_t rsX = __builtin_amdgcn_make_buffer_rsrc((void*)P.x, 0, KK * KC, 0x00020000);
    const __amdgpu_buffer_rsrc_t rsR = __builtin_amdgcn_make_buffer_rsrc((void*)(myws + WS_RECV), 0, (int)RECV_BYTES, 0x00020000);
    const unsigned scrap = (unsigned)(size_t)smem + LDS_SCRAP;
#define PF(rs, voff, soff, bits) asm volatile("s_mov_b32 m0, %0\n\ts_nop 4\n\tbuffer_load_dword %1, %2, %3 offen " bits " lds" ::"s"(scrap), "v"(voff), "s"(rs), "s"(soff) : "memory")
    const int vw = ((idx & 1) * 64 + lane / PFSH) * NN + n0 + (lane % PFSH) * (128 / PFSH);
    const int arow = m0 + (idx >> 1) * 16 + (lane & 15);
    const int vx = (rank * ML + arow) * KC + 128 * (1 + (lane >> 4) % 3), vr = arow * 128;
    int qw = 4 * rank + 2;
    int nw = 3;
#define NEXTQ(q_, n_) ((n_) < 4 ? (q_) + 1 : (n_) == 4 ? ((4 * rank + 28) & 31) : ((((q_) + 28) & 31) >> 2) == rank ? (((q_) + 24) & 31) + 1 : ((q_) + 28) & 31)
#define PF4(rs, voff, soff) asm volatile("s_mov_b32 m0, %0\n\ts_nop 0\n\tbuffer_load_dwordx4 %1, %2, %3 offen lds" ::"s"(scrap), "v"(voff), "s"(rs), "s"(soff) : "memory")
#define PFW() do { qw = NEXTQ(qw, nw); ++nw; _Pragma("unroll") for (int h_ = 0; h_ < PFSH; ++h_) { if (PFSH == 8 && !PFNB) PF4(rsW, vw, (qw << 21) + h_ * (64 / PFSH) * NN); else PF(rsW, vw, (qw << 21) + h_ * (64 / PFSH) * NN, PFBITS); } } while (0)
    const unsigned cstart = clk32();
    const int vh = ((b & 1) * 64 + lane) * NN + (((3 - (int)((size_t)P.w >> 7)) & 7) + 8 * ((b & 31) >> 1)) * 128;
    int tn = b >> 5;
    bool on = true;
#define QOF(T) __builtin_amdgcn_readfirstlane((T) < 4 ? 4 * rank + (T) : 4 * ((rank + 7 - ((T) - 4) % 7) & 7) + ((T) - 4) / 7)
    if (PFD0 && tn >= 3) { PF(rsW, vh, QOF(tn) << 21, ""); tn += 8; } else if (tn < 3) tn += 8;
    const int ve = (((b >> 1) & 1) * 64 + lane) * NN + (((63 - (int)((size_t)P.w >> 7)) & 63) + 64 * (b & 1)) * 128;
    int te = b < 128 && (b >> 2) >= 3 ? b >> 2 : NT;
    for (int j = 0; j < NBAR; ++j) {
      asm volatile("s_barrier" ::: "memory");
      if (j == 0) on = clk32() - cstart > COLDC;
      if (on && te < NT && te <= j + PFD) { PF(rsW, ve, QOF(te) << 21, ""); te = NT; }
      if (on && tn < NT && tn <= j + PFD) { PF(rsW, vh, QOF(tn) << 21, ""); tn += 8; }
    }
    asm volatile("s_waitcnt vmcnt(0)" ::: "memory");
    return;
  }

  if (wave == 4) {
    __builtin_amdgcn_s_setprio(3);
    const unsigned cstart = clk32();
#define NPUSHWG 224
    const bool pusher = P.comm && b < NPUSHWG && (P.comm != 3 || rank < 2) && (P.comm != 4 || ((rank + 1 + (b % 56) % 7) & 7) < 2);
    const int rnd = b / 56, bb = b % 56, d = 1 + bb % 7, slice = bb / 7;
    const unsigned peer = (unsigned)(rank + d) & 7u;
    v4i pay[8];
    if (pusher) {
      const unsigned char* src = P.x + ((size_t)(peer * ML + slice * 64 + (lane >> 3)) * KC + rnd * 128 + (lane & 7) * 16);
#pragma unroll
      for (int i = 0; i < 8; ++i) pay[i] = *(const v4i*)(src + i * 8 * KC);
    }
    unsigned* cnt = (unsigned*)(myws + WS_CNT) + b * 32;
    unsigned* tmo = (unsigned*)(myws + WS_TMO);
    const unsigned call = __builtin_amdgcn_readfirstlane(__hip_atomic_load(cnt, __ATOMIC_RELAXED, __HIP_MEMORY_SCOPE_AGENT)) + 1u;
    unsigned char* pws = P.ws[peer];
    unsigned char* dst = pws + WS_RECV + ((size_t)(call & 1u) * PAR + (size_t)(rank * 4 + rnd) * (ML * 128) + slice * 64 * 128 + lane * 16);
    unsigned* pflag = (unsigned*)(pws + WS_FLAG) + (lane & 7) * 256 + (rank * 8 + slice) * 4 + rnd;
    const unsigned char* myflag = myws + WS_FLAG + (b & 7) * 1024 + lane * 16;
    unsigned push_c = PUSHT0 + PUSHDT * rnd;
    const int us = lane / 7, up = (rank + 7 - lane % 7) & 7;
    const int uq = 4 * up + us;
#define CSTAMP(i)
    constexpr unsigned ALL = (1u << 28) - 1u;
    bool bad = false;
    unsigned arrived = P.comm == 1 ? 0u : ALL, used = 0u;
    int nass = 0;
    int ps = pusher ? 0 : 2;
    v4i f = {0, 0, 0, 0};
    bool polled = false, blk = false;
    int issued = 0, seq_st = 0, seq_poll = 0, nst = 0;
#define PUSH1(i) asm volatile("global_store_dwordx4 %0, %1, off " PUSHBITS "\n\ts_nop 1" ::"v"(dst + (i) * 1024), "v"(pay[i]) : "memory")
#define PUSH_PAYLOAD                                                                                                    \
    _Pragma("unroll") for (int i = 0; i < NPAYST; ++i)                                                                  \
      asm volatile("global_store_dwordx4 %0, %1, off " PUSHBITS "\n\ts_nop 1" ::"v"(dst + i * 1024), "v"(pay[i]) : "memory")
    *(LAS unsigned*)(smem + LDS_CTRL + 4 * (lane & 3)) = (lane & 3) == 0 ? call : (lane & 3) == 2 ? arrived : 0u;
    CSTAMP(0);
    asm volatile("s_waitcnt vmcnt(0)" : "+v"(pay[0]), "+v"(pay[1]), "+v"(pay[2]), "+v"(pay[3]), "+v"(pay[4]), "+v"(pay[5]), "+v"(pay[6]), "+v"(pay[7]) :: "memory");
    if (pusher && rnd == 0) { PUSH_PAYLOAD; ps = 1; issued = seq_st = 8; CSTAMP(1); }
    __hip_atomic_store(cnt, call, __ATOMIC_RELAXED, __HIP_MEMORY_SCOPE_AGENT);
    ++issued;
    for (int j = 0;; ++j) {
      const int need = j < 3 ? 0 : (j < 29 ? j - 1 : 28);
      const unsigned c0 = clk32();
      unsigned long long t0 = 0;
      bool pslot = false;
      for (;;) {
        unsigned ib; asm volatile("s_getreg_b32 %0, hwreg(HW_REG_IB_STS)" : "=s"(ib));
        const int done = issued - (int)((ib & 0xfu) | ((ib >> 18) & 0x30u));
        if (ps == 1 && done >= seq_st) {
          if (issued - seq_st >= 2) asm volatile("s_waitcnt vmcnt(2)" ::: "memory");
          else if (issued - seq_st == 1) asm volatile("s_waitcnt vmcnt(1)" ::: "memory");
          else asm volatile("s_waitcnt vmcnt(0)" ::: "memory");
          __hip_atomic_store(pflag, call, __ATOMIC_RELAXED, __HIP_MEMORY_SCOPE_SYSTEM);
          ++issued; ps = 2; if (rnd == 0) CSTAMP(2); else CSTAMP(3);
        }
        if (polled && done >= seq_poll) {
          polled = false;
          asm volatile("" : "+v"(f) :: "memory");
          const unsigned long long m0_ = __ballot((int)(f.x - call) >= 0), m1_ = __ballot((int)(f.y - call) >= 0),
                                   m2_ = __ballot((int)(f.z - call) >= 0), m3_ = __ballot((int)(f.w - call) >= 0);
          const unsigned long long mm = us == 0 ? m0_ : us == 1 ? m1_ : us == 2 ? m2_ : m3_;
          arrived = (unsigned)__ballot(lane < 28 && ((mm >> (8 * up)) & 0xffull) == 0xffull);
          *(LAS unsigned*)(smem + LDS_CTRL + 8) = arrived;
        }
        if (ps == 0 && clk32() - cstart >= push_c) {
          PUSH_PAYLOAD; issued += 8; ps = 1; seq_st = issued;
        }
        if (!polled && arrived != ALL && j >= 1 && (nass < need || !POLL1 || !pslot)) {
          pslot = true;
          asm volatile("global_load_dwordx4 %0, %1, off sc0 sc1" : "+v"(f) : "v"(myflag) : "memory");
          ++issued; seq_poll = issued; polled = true;
        }
        while (nass < need && (arrived & ~used) != 0u) {
          const int u = __builtin_ctz(arrived & ~used);
          used |= 1u << u;
          *(LAS int*)(smem + LDS_ORD + 16 + 4 * nass) = __builtin_amdgcn_readlane(uq, u);
          ++nass;
        }
        const bool idle = (ps == 2 || (ps == 0 && c0 - cstart + SPIN < push_c)) && (arrived == ALL || j < 1 || (POLL1 && pslot));
        if (nass >= need && idle) break;
        if (nass < need && !blk) { blk = true; *(LAS unsigned*)(smem + LDS_CTRL + 12) = 1u; }
        if (nass >= need && blk) { blk = false; *(LAS unsigned*)(smem + LDS_CTRL + 12) = 0u; }
        const unsigned dc = clk32() - c0;
        if (nass >= need) { if (dc > (j == 0 && rnd == 0 ? SPIN0 : SPIN)) break; }
        else if (dc > 50000u || t0 != 0) {
          const unsigned long long tn = rtc64();
          if (t0 == 0) t0 = tn;
          if (__builtin_amdgcn_readfirstlane(__hip_atomic_load(tmo, __ATOMIC_RELAXED, __HIP_MEMORY_SCOPE_AGENT)) == call || tn - t0 > DEADLINE_TICKS) {
            __hip_atomic_store(tmo, call, __ATOMIC_RELAXED, __HIP_MEMORY_SCOPE_AGENT);
            ++issued; bad = true; arrived = ALL; polled = false;
          }
          ++issued;
          __builtin_amdgcn_s_sleep(8);
        }
        __builtin_amdgcn_s_sleep(SLEEPN);
      }
      if (blk) { blk = false; *(LAS unsigned*)(smem + LDS_CTRL + 12) = 0u; }
      const bool last = (arrived == ALL && ps == 2 && !polled) || j == NBAR - 2;
      if (last) {
        const unsigned rest = ALL & ~used;
        const bool mineleft = lane < 28 && ((rest >> (lane & 31)) & 1u);
        *(LAS int*)(smem + LDS_ORD + (mineleft ? 16 + 4 * (nass + __builtin_popcount(rest & ((1u << (lane & 31)) - 1u))) : 0)) = uq;
        if (P.comm && __builtin_amdgcn_readfirstlane(__hip_atomic_load(tmo, __ATOMIC_RELAXED, __HIP_MEMORY_SCOPE_AGENT)) == call) bad = true;
        if (bad) *(LAS unsigned*)(smem + LDS_CTRL + 4) = 1u;
      }
      asm volatile("s_waitcnt lgkmcnt(0)\n\ts_barrier" ::: "memory");
      if (j == 0) __builtin_amdgcn_s_setprio(PRIO);
      if (last) {
        for (++j; j < NBAR; ++j) {
          asm volatile("s_barrier" ::: "memory");
        }
        break;
      }
    }
    asm volatile("s_waitcnt vmcnt(0)" : "+v"(f) :: "memory");
    if (ps == 1) {
      __hip_atomic_store(pflag, call, __ATOMIC_RELAXED, __HIP_MEMORY_SCOPE_SYSTEM);
    }
    return;
  }

  const int wm = wave >> 1, wn = wave & 1;
  const unsigned lds0 = (unsigned)(size_t)smem;

  const __amdgpu_buffer_rsrc_t rsW = __builtin_amdgcn_make_buffer_rsrc((void*)P.w, 0, KK * NN, 0x00020000);
  const __amdgpu_buffer_rsrc_t rsX = __builtin_amdgcn_make_buffer_rsrc((void*)P.x, 0, KK * KC, 0x00020000);
  const __amdgpu_buffer_rsrc_t rsR = __builtin_amdgcn_make_buffer_rsrc((void*)(myws + WS_RECV), 0, (int)RECV_BYTES, 0x00020000);
  const int swzA = ((lane & 7) ^ (((lane >> 4) + 4 * (wave & 1)) & 7)) * 16;
  const int voffAx = (lane >> 3) * KC + swzA;
  const int voffAr = (lane >> 3) * 128 + swzA;
  const int t16 = lane & 15, g4 = lane >> 4;
  const int voffB = (lane >> 3) * NN + (((lane & 7) ^ (((lane >> 4) & 3) | (((wave >> 1) & 1) << 2))) * 16);

#define DMAW(rs, ldsoff, voff, soff)                                                                     \
  asm volatile("s_mov_b32 m0, %0\n\ts_nop 0\n\tbuffer_load_dwordx4 %1, %2, %3 offen " WBITS " lds" ::"s"(lds0 + (ldsoff)), \
               "v"(voff), "s"(rs), "s"(soff) : "memory")
#define ISSUE_B(i, st, kb) DMAW(rsW, (st) + A_TILE + (4 * (i) + wave) * 1024, voffB, (kb) + (4 * (i) + wave) * 8 * NN)
#define ISSUE_AX(i, st, ab) DMA(rsX, (st) + (4 * (i) + wave) * 1024, voffAx, (ab) + (4 * (i) + wave) * 8 * KC)
#define ISSUE_AR(i, st, ab) DMA(rsR, (st) + (4 * (i) + wave) * 1024, voffAr, (ab) + (4 * (i) + wave) * 8 * 128)
#define STAMP(i)

  const int ax0 = (rank * ML + m0) * KC;
  const int wb0 = rank * KC * NN + n0;
#pragma unroll
  for (int T = 0; T < 3; ++T) {
#pragma unroll
    for (int i = 0; i < 4; ++i) ISSUE_B(i, T * STAGE, wb0 + T * 128 * NN);
    if (T < 2) {
#pragma unroll
      for (int i = 0; i < 8; ++i) ISSUE_AX(i, T * STAGE, ax0 + T * 128);
    }
  }
  int adA[2], adB[4];
#pragma unroll
  for (int e = 0; e < 2; ++e) adA[e] = (wm * 128 + t16) * 128 + 16 * ((4 * e + g4) ^ ((t16 >> 1) & 7));
#pragma unroll
  for (int nb = 0; nb < 4; ++nb)
    adB[nb] = A_TILE + (16 * g4 + (t16 >> 1)) * 128 + 16 * ((4 * wn + nb) ^ (((t16 >> 2) & 3) | ((g4 & 1) << 2))) + 8 * (t16 & 1);
  v4f acc[8][4];
#pragma unroll
  for (int i = 0; i < 8; ++i)
#pragma unroll
    for (int j = 0; j < 4; ++j) { acc[i][j] = (v4f){0.f, 0.f, 0.f, 0.f}; asm volatile("" : "+a"(acc[i][j])); }
  SB;
  asm volatile("s_waitcnt vmcnt(16)\n\ts_barrier" ::: "memory");
  const unsigned call = __builtin_amdgcn_readfirstlane(*(LAS unsigned*)(smem + LDS_CTRL));
  const int rbase = (int)(call & 1u) * PAR + m0 * 128;

  int st = 0;
  const int qdef = (4 * rank + 28) & 31;
  int ab = ax0 + 2 * 128;
  const LAS int* ord = (const LAS int*)(smem + LDS_ORD) + 3;
#define IA(i) do { if (AK == 0) { ISSUE_AX(i, stp, ab); } else if (AK == 1) { ISSUE_AR(i, stp, ab); } } while (0)
#define IL(i) do { if (LATE) { ISSUE_AR(i, stp, abl); } } while (0)
#define IB(i) do { if (BKD < 3) { ISSUE_B(i, st, kb); } } while (0)

  v4i aw[4][2];
  v2i bc[4][4], bn[4][4];
#define RDA(slot, mb, st) do { aw[slot][0] = *(const LAS v4i*)(smem + (st) + adA[0] + (mb) * 2048);            \
                               aw[slot][1] = *(const LAS v4i*)(smem + (st) + adA[1] + (mb) * 2048); } while (0)
#define RDB(dst, nb, i, st) \
  dst[nb][i] = __builtin_amdgcn_ds_read_tr8_b64_v2i32((LAS v2i*)(smem + (st) + adB[nb] + ((i) >> 1) * 8192 + ((i) & 1) * 1024))
#define RDB2(dst, nb, i, st) do { if (W >= 0) { RDB(dst, nb, i, st); RDB(dst, nb, (i) + 1, st); } } while (0)
#define RDAN(slot, mb) do { if (W >= 0) { RDA(slot, mb, stn); } } while (0)
#define MM(B_, mb, nb)                                                                                            \
  SB; asm volatile("v_mfma_f32_16x16x128_f8f6f4 %0, %1, %2, %0 cbsz:1 blgp:1" : "+a"(acc[mb][nb])                  \
                   : "v"(cat8(B_[nb][0], B_[nb][1], B_[nb][2], B_[nb][3])), "v"(cat8(aw[(mb) & 3][0], aw[(mb) & 3][1]))); SB
#pragma unroll
  for (int nb = 0; nb < 4; ++nb) { RDB(bc, nb, 0, 0); RDB(bc, nb, 1, 0); RDB(bc, nb, 2, 0); RDB(bc, nb, 3, 0); if (nb == 0) RDA(0, 0, 0); }
  RDA(1, 1, 0); RDA(2, 2, 0); RDA(3, 3, 0);
#define ITERX(AK_, BKD_, W_, LATE_, BC, BNX)                                                                     \
  do {                                                                                                          \
    constexpr int AK = AK_, BKD = BKD_, W = W_; constexpr bool LATE = LATE_;                                    \
    const int stn = st == 2 * STAGE ? 0 : st + STAGE;                                                           \
    const int stp = st == 0 ? 2 * STAGE : st - STAGE;                                                           \
    MM(BC, 0, 0); IA(0); MM(BC, 0, 1); MM(BC, 0, 2); MM(BC, 0, 3); RDA(0, 4, st); IA(1);                         \
    MM(BC, 1, 0); MM(BC, 1, 1); IA(2); MM(BC, 1, 2); MM(BC, 1, 3); RDA(1, 5, st);                                \
    MM(BC, 2, 0); IA(3); MM(BC, 2, 1); MM(BC, 2, 2); IA(4); MM(BC, 2, 3); RDA(2, 6, st);                         \
    MM(BC, 3, 0); MM(BC, 3, 1); IA(5); MM(BC, 3, 2); MM(BC, 3, 3); RDA(3, 7, st); IA(6);                         \
    MM(BC, 4, 0); MM(BC, 4, 1); IA(7); MM(BC, 4, 2); MM(BC, 4, 3);                                               \
    if (W == 12) asm volatile("s_waitcnt vmcnt(12) lgkmcnt(0)\n\ts_barrier" ::: "memory");                      \
    else if (W == 0) asm volatile("s_waitcnt vmcnt(0) lgkmcnt(0)\n\ts_barrier" ::: "memory");                   \
    int qv = 0, qlv = 0;                                                                                        \
    if (LATE) qlv = ord[-1];                                                                                    \
    if (BKD == 2) qv = ord[0];                                                                                  \
    ++ord;                                                                                                      \
    RDAN(0, 0);                                                                                                 \
    MM(BC, 5, 0); RDB2(BNX, 0, 0, stn); MM(BC, 5, 1); RDB2(BNX, 0, 2, stn);                                      \
    const int q = BKD == 0 ? 4 * rank + 3 : BKD == 1 ? qdef : __builtin_amdgcn_readfirstlane(qv);               \
    const int ql = LATE ? __builtin_amdgcn_readfirstlane(qlv) : 0;                                              \
    const int kb = (q << 21) + n0, abn = BKD == 0 ? ax0 + 3 * 128 : rbase + (q << 16), abl = rbase + (ql << 16); \
    if (LATE && ql != qdef) {                                                                                   \
      const int kbl = (ql << 21) + n0;                                                                          \
      ISSUE_B(0, stp, kbl); ISSUE_B(1, stp, kbl); ISSUE_B(2, stp, kbl); ISSUE_B(3, stp, kbl);                    \
    }                                                                                                           \
    IL(0); IL(1);                                                                                               \
    MM(BC, 5, 2); IL(2); RDB2(BNX, 1, 0, stn); if (!LATE) IB(0); MM(BC, 5, 3); IL(3); RDAN(1, 1);                \
    MM(BC, 6, 0); IL(4); RDB2(BNX, 1, 2, stn); MM(BC, 6, 1); IL(5); RDB2(BNX, 2, 0, stn); if (!LATE) IB(1);      \
    MM(BC, 6, 2); IL(6); RDB2(BNX, 2, 2, stn); MM(BC, 6, 3); IL(7); RDAN(2, 2); if (LATE) IB(0);                 \
    MM(BC, 7, 0); RDB2(BNX, 3, 0, stn); if (LATE) IB(1); else IB(2); MM(BC, 7, 1); RDB2(BNX, 3, 2, stn); if (LATE) IB(2); \
    MM(BC, 7, 2); IB(3); MM(BC, 7, 3); RDAN(3, 3); SB;                                                          \
    st = stn; ab = abn;                                                                                         \
  } while (0)
#define ITER(AK, BKD, W, LATE) ITERX(AK, BKD, W, LATE, bc, bn)
#define ITER2(AK, BKD, W, LATE) ITERX(AK, BKD, W, LATE, bn, bc)

  ITER(0, 0, 12, false);
  ITER2(0, 1, 12, false);
  ITER(2, 2, 0, true);
  ITER2(1, 2, 12, false);
#pragma unroll 1
  for (int t = 4; t < 28; t += 2) { ITER(1, 2, 12, false); ITER2(1, 2, 12, false); }
  ITER(1, 2, 12, false);
  ITER2(1, 3, 12, false);
  ITER(2, 3, 0, false);
  ITER2(2, 3, -1, false);

#define PIN4(i) "+a"(acc[i][0]), "+a"(acc[i][1]), "+a"(acc[i][2]), "+a"(acc[i][3])
  asm volatile("s_nop 15\n\ts_nop 7" : PIN4(7), PIN4(6), PIN4(5), PIN4(4));
  asm volatile("" : PIN4(3), PIN4(2), PIN4(1), PIN4(0));
  SB;
  STAMP(2);
  const float scale = P.sx[0] * P.sw[0];
  const bool bad = *(LAS unsigned*)(smem + LDS_CTRL + 4) != 0u;
#define FIN(y_) (bad ? __builtin_nanf("") : ((y_) < 0.f ? 0.f : (y_)))
  asm volatile("s_waitcnt lgkmcnt(0)\n\ts_barrier" ::: "memory");
  LAS char* img = smem + wave * 32768;
  float* obase = P.out + (size_t)(m0 + wm * 128 + (lane >> 4)) * NN + n0 + wn * 64 + (lane & 15) * 4;
#pragma unroll
  for (int mq = 0; mq < 4; ++mq) {
#pragma unroll
    for (int mh = 0; mh < 2; ++mh)
#pragma unroll
      for (int nb = 0; nb < 4; ++nb) {
        v4f v;
#pragma unroll
        for (int e = 0; e < 4; ++e) { const float y = acc[2 * mq + mh][nb][e] * scale; v[e] = FIN(y); }
        *(LAS v4f*)(img + (mq * 32 + mh * 16 + t16) * 256 + (((nb * 4 + g4) ^ (t16 & 7)) * 16)) = v;
      }
#pragma unroll
    for (int i = 0; i < 8; ++i) {
      const int row = mq * 32 + i * 4 + (lane >> 4);
      const v4f v = *(const LAS v4f*)(img + row * 256 + (((lane & 15) ^ (row & 7)) * 16));
      float* ptr = obase + (size_t)(mq * 32 + i * 4) * NN;
      asm volatile("global_store_dwordx4 %0, %1, off " OUTBITS "\n\ts_nop 1" ::"v"(ptr), "v"(v) : "memory");
    }
  }
}

#define HIP_CHECK(expr) do { hipError_t _e = (expr); if (_e != hipSuccess) { \
    fprintf(stderr, "HIP error %s at %s:%d\n", hipGetErrorString(_e), __FILE__, __LINE__); \
    return; } } while (0)

extern "C" void kernel_launch(int rank, int nranks, ncclComm_t comm, hipStream_t stream,
                              void* const* d_in, const int* in_sizes, int n_in,
                              void* d_out, int out_size, const int* in_dtypes, int out_dtype,
                              void* const* d_out_all, void* const* d_ws_all, size_t ws_bytes) {
  if (nranks != NR || (unsigned)rank >= (unsigned)NR || n_in != 4 || ws_bytes < WS_NEED || out_size != ML * NN ||
      in_sizes[0] != KK * KC || in_sizes[1] != KK * NN) {
    fprintf(stderr, "kernel_launch: unexpected problem shape\n");
    return;
  }
  static bool configured = false;
  if (!configured) {
    HIP_CHECK(hipFuncSetAttribute((const void*)fused_a2a_gemm, hipFuncAttributeMaxDynamicSharedMemorySize, LDS_BYTES));
    configured = true;
  }
  Params P{};
  P.x = (const unsigned char*)d_in[0];
  P.w = (const unsigned char*)d_in[1];
  P.sx = (const float*)d_in[2];
  P.sw = (const float*)d_in[3];
  P.out = (float*)d_out;
  for (int p = 0; p < NR; ++p) P.ws[p] = (unsigned char*)d_ws_all[p];
  P.rank = rank;
  P.myws = P.ws[rank];
  P.comm = 1;
  fused_a2a_gemm<<<256, NTHREADS, LDS_BYTES, stream>>>(P);
  HIP_CHECK(hipGetLastError());
}
```

```cpp
#include <hip/hip_runtime.h>
#include <rccl/rccl.h>
#include <cstdio>

#define LAS __attribute__((address_space(3)))
typedef int v2i __attribute__((ext_vector_type(2)));
typedef int v4i __attribute__((ext_vector_type(4)));
typedef int v8i __attribute__((ext_vector_type(8)));
typedef float v4f __attribute__((ext_vector_type(4)));

constexpr int NR = 8;
constexpr int ML = 512, NN = 16384, KK = 4096, KC = 512;
constexpr int BM = 256, BN = 128, BK = 128, NT = KK / BK;
constexpr int NTHREADS = 384;
constexpr int A_TILE = BM * BK, B_TILE = BK * BN, STAGE = A_TILE + B_TILE, NSTAGE = 3;
constexpr int NBAR = NT;
constexpr int LDS_CTRL = NSTAGE * STAGE;
constexpr int LDS_ORD = LDS_CTRL + 64;
constexpr int LDS_SCRAP = LDS_ORD + 128;
constexpr int LDS_BYTES = LDS_SCRAP + 256;
constexpr size_t WS_CNT = 0;
constexpr size_t WS_TMO = 32768;
constexpr size_t WS_FLAG = 65536;
constexpr size_t WS_RECV = 1u << 20;
constexpr int PAR = NR * KC * ML + (512 << 10);
constexpr size_t RECV_BYTES = (size_t)PAR + NR * KC * ML;
constexpr size_t WS_NEED = WS_RECV + RECV_BYTES;
constexpr unsigned long long DEADLINE_TICKS = 50000000ull;
constexpr unsigned PUSH_T0 = 800, PUSH_DT = 2800;
constexpr unsigned SPIN = 600, SPIN0 = 200;
constexpr unsigned COLD_CYC = 4500;
constexpr unsigned WINMASK = (1u << 16) - 1u;
constexpr int PF_DIST = 16;

struct Params {
  const unsigned char* x;
  const unsigned char* w;
  const float* sx;
  const float* sw;
  float* out;
  unsigned char* myws;
  int rank;
  int comm;
  unsigned char* ws[NR];
};

#define SB __builtin_amdgcn_sched_barrier(0)
#define DMA(rs, ldsoff, voff, soff)                                                                              \
  asm volatile("s_mov_b32 m0, %0\n\ts_nop 0\n\tbuffer_load_dwordx4 %1, %2, %3 offen lds" ::"s"(lds0 + (ldsoff)), \
               "v"(voff), "s"(rs), "s"(soff) : "memory")

__device__ __forceinline__ v8i cat8(v4i a, v4i b) { return __builtin_shufflevector(a, b, 0, 1, 2, 3, 4, 5, 6, 7); }
__device__ __forceinline__ v8i cat8(v2i a, v2i b, v2i c, v2i d) {
  v4i lo = __builtin_shufflevector(a, b, 0, 1, 2, 3), hi = __builtin_shufflevector(c, d, 0, 1, 2, 3);
  return cat8(lo, hi);
}
__device__ __forceinline__ unsigned clk32() { unsigned long long t; asm volatile("s_memtime %0\n\ts_waitcnt lgkmcnt(0)" : "=s"(t) :: "memory"); return (unsigned)t; }
__device__ __forceinline__ unsigned long long rtc64() { unsigned long long t; asm volatile("s_memrealtime %0\n\ts_waitcnt lgkmcnt(0)" : "=s"(t) :: "memory"); return t; }

__global__ __launch_bounds__(NTHREADS, 2) void fused_a2a_gemm(Params P) {
  extern __shared__ __attribute__((aligned(16))) char smem_g[];
  LAS char* smem = (LAS char*)smem_g;
  const int tid = threadIdx.x, lane = tid & 63;
  const int wave = __builtin_amdgcn_readfirstlane(tid >> 6);
  const int b = blockIdx.x;
  const int rank = P.rank;
  unsigned char* myws = P.myws;
  const int xcd = b & 7, idx = b >> 3;
  const int ntile = (idx >> 1) * 8 + ((xcd + (idx >> 1)) & 7);
  const int n0 = ntile * BN, m0 = (idx & 1) * BM;

  if (wave == 5) {
    const __amdgpu_buffer_rsrc_t rsW = __builtin_amdgcn_make_buffer_rsrc((void*)P.w, 0, KK * NN, 0x00020000);
    const unsigned scrap = (unsigned)(size_t)smem + LDS_SCRAP;
#define QOF(T) __builtin_amdgcn_readfirstlane((T) < 4 ? 4 * rank + (T) : 4 * ((rank + 7 - ((T) - 4) % 7) & 7) + ((T) - 4) / 7)
#define PF(voff, T) asm volatile("s_mov_b32 m0, %0\n\ts_nop 4\n\tbuffer_load_dword %1, %2, %3 offen lds" ::"s"(scrap), "v"(voff), "s"(rsW), "s"(QOF(T) << 21) : "memory")
    const unsigned cstart = clk32();
    const int wline = (int)((size_t)P.w >> 7);
    const int vh = ((b & 1) * 64 + lane) * NN + (((3 - wline) & 7) + 8 * ((b & 31) >> 1)) * 128;
    const int ve = (((b >> 1) & 1) * 64 + lane) * NN + (((63 - wline) & 63) + 64 * (b & 1)) * 128;
    int tn = b >> 5, te = b < 128 && (b >> 2) >= 3 ? b >> 2 : NT;
    if (tn >= 3) PF(vh, tn);
    tn += 8;
    bool on = true;
    for (int j = 0; j < NBAR; ++j) {
      asm volatile("s_barrier" ::: "memory");
      if (j == 0) on = clk32() - cstart > COLD_CYC;
      if (on && te < NT && te <= j + PF_DIST) { PF(ve, te); te = NT; }
      if (on && tn < NT && tn <= j + PF_DIST) { PF(vh, tn); tn += 8; }
    }
    asm volatile("s_waitcnt vmcnt(0)" ::: "memory");
    return;
  }

  if (wave == 4) {
    __builtin_amdgcn_s_setprio(3);
    const unsigned cstart = clk32();
    const bool pusher = P.comm && b < 224;
    const int rnd = b / 56, bb = b % 56, slice = bb / 7;
    const unsigned peer = (unsigned)(rank + 1 + bb % 7) & 7u;
    v4i pay[8];
    if (pusher) {
      const unsigned char* src = P.x + ((size_t)(peer * ML + slice * 64 + (lane >> 3)) * KC + rnd * 128 + (lane & 7) * 16);
#pragma unroll
      for (int i = 0; i < 8; ++i) pay[i] = *(const v4i*)(src + i * 8 * KC);
    }
    unsigned* cnt = (unsigned*)(myws + WS_CNT) + b * 32;
    unsigned* tmo = (unsigned*)(myws + WS_TMO);
    const unsigned call = __builtin_amdgcn_readfirstlane(__hip_atomic_load(cnt, __ATOMIC_RELAXED, __HIP_MEMORY_SCOPE_AGENT)) + 1u;
    unsigned char* pws = P.ws[peer];
    unsigned char* dst = pws + WS_RECV + ((size_t)(call & 1u) * PAR + (size_t)(rank * 4 + rnd) * (ML * 128) + slice * 64 * 128 + lane * 16);
    unsigned* pflag = (unsigned*)(pws + WS_FLAG) + ((slice >> 2) * 8 + (lane & 7)) * 128 + (rank * 4 + (slice & 3)) * 4 + rnd;
    const unsigned char* myflag = myws + WS_FLAG + ((idx & 1) * 8 + (b & 7)) * 512 + (lane & 31) * 16;
    const int us = lane / 7, up = (rank + 7 - lane % 7) & 7;
    const int uq = 4 * up + us;
    constexpr unsigned ALL = (1u << 28) - 1u;
    bool bad = false, polled = false;
    unsigned arrived = P.comm ? 0u : ALL, used = 0u;
    int nass = 0;
    int ps = pusher ? 0 : 2;
    v4i f = {0, 0, 0, 0};
    int issued = 0, seq_st = 0, seq_poll = 0;
#define PUSH_PAYLOAD                                                                                                    \
    _Pragma("unroll") for (int i = 0; i < 8; ++i)                                                                       \
      asm volatile("global_store_dwordx4 %0, %1, off sc0 sc1 nt\n\ts_nop 1" ::"v"(dst + i * 1024), "v"(pay[i]) : "memory")
#define CHECK_TMO                                                                                                       \
    do {                                                                                                                \
      if (P.comm && __builtin_amdgcn_readfirstlane(__hip_atomic_load(tmo, __ATOMIC_RELAXED, __HIP_MEMORY_SCOPE_AGENT)) == call) bad = true; \
      if (bad) *(LAS unsigned*)(smem + LDS_CTRL + 4) = 1u;                                                              \
    } while (0)
    *(LAS unsigned*)(smem + LDS_CTRL + 4 * (lane & 1)) = (lane & 1) ? 0u : call;
    asm volatile("s_waitcnt vmcnt(0)" : "+v"(pay[0]), "+v"(pay[1]), "+v"(pay[2]), "+v"(pay[3]), "+v"(pay[4]), "+v"(pay[5]), "+v"(pay[6]), "+v"(pay[7]) :: "memory");
    const unsigned push_c = clk32() - cstart + PUSH_T0 + PUSH_DT * rnd;
    if (pusher && rnd == 0) { PUSH_PAYLOAD; ps = 1; issued = seq_st = 8; }
    __hip_atomic_store(cnt, call, __ATOMIC_RELAXED, __HIP_MEMORY_SCOPE_AGENT);
    ++issued;
    for (int j = 0;; ++j) {
      const int need = j < 3 ? 0 : (j < 29 ? j - 1 : 28);
      const unsigned c0 = clk32();
      unsigned long long t0 = 0;
      bool pslot = false;
      for (;;) {
        unsigned ib; asm volatile("s_getreg_b32 %0, hwreg(HW_REG_IB_STS)" : "=s"(ib));
        const int done = issued - (int)((ib & 0xfu) | ((ib >> 18) & 0x30u));
        if (ps == 1 && done >= seq_st) {
          if (issued - seq_st >= 2) asm volatile("s_waitcnt vmcnt(2)" ::: "memory");
          else if (issued - seq_st == 1) asm volatile("s_waitcnt vmcnt(1)" ::: "memory");
          else asm volatile("s_waitcnt vmcnt(0)" ::: "memory");
          __hip_atomic_store(pflag, call, __ATOMIC_RELAXED, __HIP_MEMORY_SCOPE_SYSTEM);
          ++issued; ps = 2;
        }
        if (polled && done >= seq_poll) {
          polled = false;
          asm volatile("" : "+v"(f) :: "memory");
          const unsigned long long m0_ = __ballot((int)(f.x - call) >= 0), m1_ = __ballot((int)(f.y - call) >= 0),
                                   m2_ = __ballot((int)(f.z - call) >= 0), m3_ = __ballot((int)(f.w - call) >= 0);
          const unsigned long long mm = us == 0 ? m0_ : us == 1 ? m1_ : us == 2 ? m2_ : m3_;
          arrived = (unsigned)__ballot(lane < 28 && ((mm >> (4 * up)) & 0xfull) == 0xfull);
        }
        if (ps == 0 && clk32() - cstart >= push_c) { PUSH_PAYLOAD; issued += 8; ps = 1; seq_st = issued; }
        if (!polled && arrived != ALL && j >= 1 && (nass < need || !pslot)) {
          asm volatile("global_load_dwordx4 %0, %1, off sc0 sc1" : "+v"(f) : "v"(myflag) : "memory");
          ++issued; seq_poll = issued; polled = true; pslot = true;
        }
        while (nass < need) {
          const unsigned cand = arrived & ~used & (WINMASK << __builtin_ctz(~used));
          if (cand == 0u) break;
          const int u = __builtin_ctz(cand);
          used |= 1u << u;
          *(LAS int*)(smem + LDS_ORD + 16 + 4 * nass) = __builtin_amdgcn_readlane(uq, u);
          ++nass;
        }
        const bool idle = (ps == 2 || (ps == 0 && c0 - cstart + SPIN < push_c)) && (arrived == ALL || j < 1 || pslot);
        if (nass >= need && idle) break;
        const unsigned dc = clk32() - c0;
        if (nass >= need) { if (dc > (j == 0 && rnd == 0 ? SPIN0 : SPIN)) break; }
        else if (dc > 50000u || t0 != 0) {
          const unsigned long long tn = rtc64();
          if (t0 == 0) t0 = tn;
          if (__builtin_amdgcn_readfirstlane(__hip_atomic_load(tmo, __ATOMIC_RELAXED, __HIP_MEMORY_SCOPE_AGENT)) == call || tn - t0 > DEADLINE_TICKS) {
            __hip_atomic_store(tmo, call, __ATOMIC_RELAXED, __HIP_MEMORY_SCOPE_AGENT);
            ++issued; bad = true; arrived = ALL; polled = false;
          }
          ++issued;
          __builtin_amdgcn_s_sleep(8);
        }
        __builtin_amdgcn_s_sleep(1);
      }
      const bool last = (arrived == ALL && ps == 2 && !polled) || j == NBAR - 1;
      if (last) {
        const unsigned rest = ALL & ~used;
        const bool mine = lane < 28 && ((rest >> (lane & 31)) & 1u);
        *(LAS int*)(smem + LDS_ORD + (mine ? 16 + 4 * (nass + __builtin_popcount(rest & ((1u << (lane & 31)) - 1u))) : 0)) = uq;
        if (j >= NBAR - 2) CHECK_TMO;
      }
      asm volatile("s_waitcnt lgkmcnt(0)\n\ts_barrier" ::: "memory");
      if (j == 0) __builtin_amdgcn_s_setprio(0);
      if (last) {
        if (j < NBAR - 2) {
          asm volatile("global_load_dwordx4 %0, %1, off sc1\n\ts_barrier\n\ts_waitcnt vmcnt(0)" : "+v"(f) : "v"(tmo) : "memory");
          ++j;
          if (P.comm && (unsigned)__builtin_amdgcn_readfirstlane(f.x) == call) bad = true;
          if (bad) *(LAS unsigned*)(smem + LDS_CTRL + 4) = 1u;
        }
        for (++j; j < NBAR; ++j) asm volatile("s_waitcnt lgkmcnt(0)\n\ts_barrier" ::: "memory");
        break;
      }
    }
    asm volatile("s_waitcnt vmcnt(0)" : "+v"(f) :: "memory");
    if (ps == 1) __hip_atomic_store(pflag, call, __ATOMIC_RELAXED, __HIP_MEMORY_SCOPE_SYSTEM);
    return;
  }

  const int wm = wave >> 1, wn = wave & 1;
  const unsigned lds0 = (unsigned)(size_t)smem;
  const __amdgpu_buffer_rsrc_t rsW = __builtin_amdgcn_make_buffer_rsrc((void*)P.w, 0, KK * NN, 0x00020000);
  const __amdgpu_buffer_rsrc_t rsX = __builtin_amdgcn_make_buffer_rsrc((void*)P.x, 0, KK * KC, 0x00020000);
  const __amdgpu_buffer_rsrc_t rsR = __builtin_amdgcn_make_buffer_rsrc((void*)(myws + WS_RECV), 0, (int)RECV_BYTES, 0x00020000);
  const int swzA = ((lane & 7) ^ (((lane >> 4) + 4 * (wave & 1)) & 7)) * 16;
  const int voffAx = (lane >> 3) * KC + swzA;
  const int voffAr = (lane >> 3) * 128 + swzA;
  const int t16 = lane & 15, g4 = lane >> 4;
  const int voffB = (lane >> 3) * NN + (((lane & 7) ^ (((lane >> 4) & 3) | (((wave >> 1) & 1) << 2))) * 16);
#define ISSUE_B(i, st, kb) DMA(rsW, (st) + A_TILE + (4 * (i) + wave) * 1024, voffB, (kb) + (4 * (i) + wave) * 8 * NN)
#define ISSUE_AX(i, st, ab) DMA(rsX, (st) + (4 * (i) + wave) * 1024, voffAx, (ab) + (4 * (i) + wave) * 8 * KC)
#define ISSUE_AR(i, st, ab) DMA(rsR, (st) + (4 * (i) + wave) * 1024, voffAr, (ab) + (4 * (i) + wave) * 8 * 128)

  const int ax0 = (rank * ML + m0) * KC;
  const int wb0 = rank * KC * NN + n0;
#pragma unroll
  for (int T = 0; T < 3; ++T) {
#pragma unroll
    for (int i = 0; i < 4; ++i) ISSUE_B(i, T * STAGE, wb0 + T * 128 * NN);
#pragma unroll
    for (int i = 0; i < 8; ++i) ISSUE_AX(i, T * STAGE, ax0 + T * 128);
  }
  float sxv, swv;
  asm volatile("s_load_dword %0, %2, 0x0\n\ts_load_dword %1, %3, 0x0\n\ts_waitcnt lgkmcnt(0)" : "=&s"(sxv), "=&s"(swv) : "s"(P.sx), "s"(P.sw) : "memory");
  float* const outp = P.out;
  asm volatile("" ::"s"(outp));
  int adA[2], adB[4];
#pragma unroll
  for (int e = 0; e < 2; ++e) adA[e] = (wm * 128 + t16) * 128 + 16 * ((4 * e + g4) ^ ((t16 >> 1) & 7));
#pragma unroll
  for (int nb = 0; nb < 4; ++nb)
    adB[nb] = A_TILE + (16 * g4 + (t16 >> 1)) * 128 + 16 * ((4 * wn + nb) ^ (((t16 >> 2) & 3) | ((g4 & 1) << 2))) + 8 * (t16 & 1);
  v4f acc[8][4];
#pragma unroll
  for (int i = 0; i < 8; ++i)
#pragma unroll
    for (int j = 0; j < 4; ++j) { acc[i][j] = (v4f){0.f, 0.f, 0.f, 0.f}; asm volatile("" : "+a"(acc[i][j])); }
  SB;
  asm volatile("s_waitcnt vmcnt(24)\n\ts_barrier" ::: "memory");
  const unsigned call = __builtin_amdgcn_readfirstlane(*(LAS unsigned*)(smem + LDS_CTRL));
  const int rbase = (int)(call & 1u) * PAR + m0 * 128;

  int st = 0;
  const int qdef = (4 * rank + 28) & 31;
  int ab = 0;
  const LAS int* ord = (const LAS int*)(smem + LDS_ORD) + 3;
#define IA(i) do { if (AK == 0) { ISSUE_AX(i, stp, ab); } else if (AK == 1) { ISSUE_AR(i, stp, ab); } } while (0)
#define IL(i) do { if (LATE) { ISSUE_AR(i, stp, abl); } } while (0)
#define IB(i) do { if (BKD < 3) { ISSUE_B(i, st, kb); } } while (0)
  v4i aw[4][2];
  v2i bc[4][4], bn[4][4];
#define RDA(slot, mb, st) do { aw[slot][0] = *(const LAS v4i*)(smem + (st) + adA[0] + (mb) * 2048);            \
                               aw[slot][1] = *(const LAS v4i*)(smem + (st) + adA[1] + (mb) * 2048); } while (0)
#define RDB(dst, nb, i, st) \
  dst[nb][i] = __builtin_amdgcn_ds_read_tr8_b64_v2i32((LAS v2i*)(smem + (st) + adB[nb] + ((i) >> 1) * 8192 + ((i) & 1) * 1024))
#define RDB2(dst, nb, i, st) do { if (W >= 0) { RDB(dst, nb, i, st); RDB(dst, nb, (i) + 1, st); } } while (0)
#define RDAN(slot, mb) do { if (W >= 0) { RDA(slot, mb, stn); } } while (0)
#define MM(B_, mb, nb)                                                                                            \
  SB; asm volatile("v_mfma_f32_16x16x128_f8f6f4 %0, %1, %2, %0 cbsz:1 blgp:1" : "+a"(acc[mb][nb])                  \
                   : "v"(cat8(B_[nb][0], B_[nb][1], B_[nb][2], B_[nb][3])), "v"(cat8(aw[(mb) & 3][0], aw[(mb) & 3][1]))); SB
#pragma unroll
  for (int nb = 0; nb < 4; ++nb) { RDB(bc, nb, 0, 0); RDB(bc, nb, 1, 0); RDB(bc, nb, 2, 0); RDB(bc, nb, 3, 0); if (nb == 0) RDA(0, 0, 0); }
  RDA(1, 1, 0); RDA(2, 2, 0); RDA(3, 3, 0);
#define ITERX(AK_, BKD_, W_, LATE_, BC, BNX)                                                                     \
  do {                                                                                                          \
    constexpr int AK = AK_, BKD = BKD_, W = W_; constexpr bool LATE = LATE_;                                    \
    const int stn = st == 2 * STAGE ? 0 : st + STAGE;                                                           \
    const int stp = st == 0 ? 2 * STAGE : st - STAGE;                                                           \
    MM(BC, 0, 0); IA(0); MM(BC, 0, 1); MM(BC, 0, 2); MM(BC, 0, 3); RDA(0, 4, st); IA(1);                         \
    MM(BC, 1, 0); MM(BC, 1, 1); IA(2); MM(BC, 1, 2); MM(BC, 1, 3); RDA(1, 5, st);                                \
    MM(BC, 2, 0); IA(3); MM(BC, 2, 1); MM(BC, 2, 2); IA(4); MM(BC, 2, 3); RDA(2, 6, st);                         \
    MM(BC, 3, 0); MM(BC, 3, 1); IA(5); MM(BC, 3, 2); MM(BC, 3, 3); RDA(3, 7, st); IA(6);                         \
    MM(BC, 4, 0); MM(BC, 4, 1); IA(7); MM(BC, 4, 2); MM(BC, 4, 3);                                               \
    if (W == 12) asm volatile("s_waitcnt vmcnt(12) lgkmcnt(0)\n\ts_barrier" ::: "memory");                      \
    else if (W == 0) asm volatile("s_waitcnt vmcnt(0) lgkmcnt(0)\n\ts_barrier" ::: "memory");                   \
    int qv = 0, qlv = 0;                                                                                        \
    if (LATE) qlv = ord[-1];                                                                                    \
    if (BKD == 2) qv = ord[0];                                                                                  \
    ++ord;                                                                                                      \
    RDAN(0, 0);                                                                                                 \
    MM(BC, 5, 0); RDB2(BNX, 0, 0, stn); MM(BC, 5, 1); RDB2(BNX, 0, 2, stn);                                      \
    const int q = BKD == 0 ? 4 * rank + 3 : BKD == 1 ? qdef : __builtin_amdgcn_readfirstlane(qv);               \
    const int ql = LATE ? __builtin_amdgcn_readfirstlane(qlv) : 0;                                              \
    const int kb = (q << 21) + n0, abn = BKD == 0 ? ax0 + 3 * 128 : rbase + (q << 16), abl = rbase + (ql << 16); \
    if (LATE && ql != qdef) {                                                                                   \
      const int kbl = (ql << 21) + n0;                                                                          \
      ISSUE_B(0, stp, kbl); ISSUE_B(1, stp, kbl); ISSUE_B(2, stp, kbl); ISSUE_B(3, stp, kbl);                    \
    }                                                                                                           \
    IL(0); IL(1);                                                                                               \
    MM(BC, 5, 2); IL(2); RDB2(BNX, 1, 0, stn); if (!LATE) IB(0); MM(BC, 5, 3); IL(3); RDAN(1, 1);                \
    MM(BC, 6, 0); IL(4); RDB2(BNX, 1, 2, stn); MM(BC, 6, 1); IL(5); RDB2(BNX, 2, 0, stn); if (!LATE) IB(1);      \
    MM(BC, 6, 2); IL(6); RDB2(BNX, 2, 2, stn); MM(BC, 6, 3); IL(7); RDAN(2, 2); if (LATE) IB(0);                 \
    MM(BC, 7, 0); RDB2(BNX, 3, 0, stn); if (LATE) IB(1); else IB(2); MM(BC, 7, 1); RDB2(BNX, 3, 2, stn); if (LATE) IB(2); \
    MM(BC, 7, 2); IB(3); MM(BC, 7, 3); RDAN(3, 3); SB;                                                          \
    st = stn; ab = abn;                                                                                         \
  } while (0)
#define ITER(AK, BKD, W, LATE) ITERX(AK, BKD, W, LATE, bc, bn)
#define ITER2(AK, BKD, W, LATE) ITERX(AK, BKD, W, LATE, bn, bc)

  ITER(2, 0, 12, false);
  ITER2(0, 1, 12, false);
  ITER(2, 2, 0, true);
  ITER2(1, 2, 12, false);
#pragma unroll 1
  for (int t = 4; t < 28; t += 2) { ITER(1, 2, 12, false); ITER2(1, 2, 12, false); }
  ITER(1, 2, 12, false);
  ITER2(1, 3, 12, false);
  ITER(2, 3, 0, false);
  ITER2(2, 3, -1, false);

#define PIN4(i) "+a"(acc[i][0]), "+a"(acc[i][1]), "+a"(acc[i][2]), "+a"(acc[i][3])
  asm volatile("s_nop 15\n\ts_nop 7" : PIN4(7), PIN4(6), PIN4(5), PIN4(4));
  asm volatile("" : PIN4(3), PIN4(2), PIN4(1), PIN4(0));
  SB;
  const bool bad = *(LAS unsigned*)(smem + LDS_CTRL + 4) != 0u;
  const float scale = bad ? __builtin_nanf("") : sxv * swv;
  LAS char* img = smem + (wave >> 1) * (2 * STAGE) + (wave & 1) * 16384;
  float* obase = outp + (size_t)(m0 + wm * 128 + (lane >> 4)) * NN + n0 + wn * 64 + (lane & 15) * 4;
#pragma unroll
  for (int mq = 0; mq < 4; ++mq) {
#pragma unroll
    for (int mh = 0; mh < 2; ++mh)
#pragma unroll
      for (int nb = 0; nb < 4; ++nb)
        *(LAS v4f*)(img + ((mq & 1) * 32 + mh * 16 + t16) * 256 + (((nb * 4 + g4) ^ (t16 & 7)) * 16)) = acc[2 * mq + mh][nb];
#pragma unroll
    for (int i = 0; i < 8; ++i) {
      const int row = (mq & 1) * 32 + i * 4 + (lane >> 4);
      v4f v = *(const LAS v4f*)(img + row * 256 + (((lane & 15) ^ (row & 7)) * 16));
#pragma unroll
      for (int e = 0; e < 4; ++e) { const float y = v[e] * scale; v[e] = y < 0.f ? 0.f : y; }
      float* ptr = obase + (size_t)(mq * 32 + i * 4) * NN;
      asm volatile("global_store_dwordx4 %0, %1, off sc0 sc1\n\ts_nop 1" ::"v"(ptr), "v"(v) : "memory");
    }
  }
}

#define HIP_CHECK(expr) do { hipError_t _e = (expr); if (_e != hipSuccess) { \
    fprintf(stderr, "HIP error %s at %s:%d\n", hipGetErrorString(_e), __FILE__, __LINE__); \
    return; } } while (0)

extern "C" void kernel_launch(int rank, int nranks, ncclComm_t comm, hipStream_t stream,
                              void* const* d_in, const int* in_sizes, int n_in,
                              void* d_out, int out_size, const int* in_dtypes, int out_dtype,
                              void* const* d_out_all, void* const* d_ws_all, size_t ws_bytes) {
  if (nranks != NR || (unsigned)rank >= (unsigned)NR || n_in != 4 || ws_bytes < WS_NEED || out_size != ML * NN ||
      in_sizes[0] != KK * KC || in_sizes[1] != KK * NN) {
    fprintf(stderr, "kernel_launch: unexpected problem shape\n");
    return;
  }
  static bool configured = false;
  if (!configured) {
    HIP_CHECK(hipFuncSetAttribute((const void*)fused_a2a_gemm, hipFuncAttributeMaxDynamicSharedMemorySize, LDS_BYTES));
    configured = true;
  }
  Params P{};
  P.x = (const unsigned char*)d_in[0];
  P.w = (const unsigned char*)d_in[1];
  P.sx = (const float*)d_in[2];
  P.sw = (const float*)d_in[3];
  P.out = (float*)d_out;
  for (int p = 0; p < NR; ++p) P.ws[p] = (unsigned char*)d_ws_all[p];
  P.rank = rank;
  P.myws = P.ws[rank];
  P.comm = 1;
  fused_a2a_gemm<<<256, NTHREADS, LDS_BYTES, stream>>>(P);
  HIP_CHECK(hipGetLastError());
}
```
